```python
import numpy as np

kernel_name = "gpu_ag_gemm_m4096_k4096_n16384_e4m3e4m3_dqrelu"


ABSMAX_RTOL = 0.03

COMPUTE_DTYPE = "float32"
EPILOGUE = "dequant_relu"
IN_QUANT = None
OUT_QUANT = None
IN_FP8 = "e4m3e4m3"
IN_DTYPES = {"x": "e4m3", "w_mat": "e4m3",
             "scale_x": "float32", "scale_w": "float32"}
WORLD_SIZE = 8

def _snap_fp8(x, mant_bits, max_abs, min_normal):
    a = np.abs(x.astype(np.float64))
    _m, _e = np.frexp(a)
    q = np.where(a >= min_normal, np.ldexp(1.0, _e - 1 - mant_bits),
                 min_normal * 2.0 ** -mant_bits)
    snapped = np.minimum(np.round(a / q) * q, max_abs)
    return (np.sign(x) * snapped).astype(np.float32)

def setup_inputs():

    x = _snap_fp8(np.random.randn(4096, 4096).astype(np.float32), 3, 448.0, 2.0 ** -6)
    w_mat = _snap_fp8(np.random.randn(4096, 16384).astype(np.float32), 3, 448.0, 2.0 ** -6)
    scale_x = (0.02 * (1.0 + np.random.rand(1))).astype(np.float32)
    scale_w = (0.02 * (1.0 + np.random.rand(1))).astype(np.float32)
    return {"x": x, "w_mat": w_mat, "scale_x": scale_x, "scale_w": scale_w}

def reference(x, w_mat, scale_x, scale_w):
    acc = (x @ w_mat).astype(np.float32)
    y = (acc * (scale_x * scale_w)).astype(np.float32)
    return np.maximum(y, 0.0).astype(np.float32)
```

```cpp
#include <hip/hip_runtime.h>
#include <rccl/rccl.h>
#include <cstdio>
#include <cstdint>

#define HIP_CHECK(expr) do { hipError_t _e = (expr); if (_e != hipSuccess) { \
    fprintf(stderr, "HIP error %s at %s:%d\n", hipGetErrorString(_e), __FILE__, __LINE__); \
    return; } } while (0)

typedef int   v8i __attribute__((ext_vector_type(8)));
typedef int   v4i __attribute__((ext_vector_type(4)));
typedef int   v2i __attribute__((ext_vector_type(2)));
typedef float v4f __attribute__((ext_vector_type(4)));
#define LAS __attribute__((address_space(3)))
#define GAS __attribute__((address_space(1)))

namespace {
constexpr int NR = 8;
constexpr int KDIM = 4096;
constexpr int NL = 2048;
constexpr int MS = 512;
constexpr int NSTEP = 32;
constexpr int NWG = 256;
constexpr size_t XBYTES = (size_t)MS * KDIM;
constexpr size_t FLAG_OFF = 2 * XBYTES;
constexpr size_t CNT_OFF = FLAG_OFF + 65536;
constexpr size_t TMO_OFF = CNT_OFF + 65536;
constexpr size_t WS_NEED = TMO_OFF + 4096;
constexpr unsigned long long DEADLINE_TICKS = 50000000ull;
constexpr int WSLOT = 16384, NSLOT = 8;

struct Params {
    const char* x;
    const char* w;
    const float* sx;
    const float* sw;
    float* out;
    char* ws[NR];
    int rank;
};

__device__ __forceinline__ void glds16(unsigned voff, const char* sbase, unsigned lds_dst) {
    unsigned keep;
    asm volatile("s_mov_b32 %0, m0\n\ts_mov_b32 m0, %3\n\ts_nop 0\n\tglobal_load_lds_dwordx4 %1, %2\n\ts_mov_b32 m0, %0"
                 : "=&s"(keep) : "v"(voff), "s"(sbase), "s"(lds_dst) : "memory");
}

__global__ void __launch_bounds__(512, 2) ag_gemm_kernel(Params p) {
    __shared__ __attribute__((aligned(16))) char smem[NSLOT * WSLOT + 64];
    const int tid = threadIdx.x, lane = tid & 63, b = blockIdx.x;
    const int wid = __builtin_amdgcn_readfirstlane(tid >> 6);
    char* myws = p.ws[p.rank & 7];

    unsigned* cntp = (unsigned*)(myws + CNT_OFF) + b;
    const unsigned call = __builtin_amdgcn_readfirstlane(__hip_atomic_load(cntp, __ATOMIC_RELAXED, __HIP_MEMORY_SCOPE_AGENT)) + 1u;
    v4i xv;
    {
        const int mt = b >> 7, ks = (b >> 2) & 31, w = (b & 3) * 2 + (wid >> 2), rb = (wid >> 1) & 1, j = wid & 1;
        const int row = mt * 256 + w * 32 + rb * 16 + (lane & 15), kk = ks * 128 + 32 * (lane >> 4) + 16 * j;
        xv = *(const v4i*)(p.x + (size_t)row * KDIM + kk);
    }
    if (wid == 0) __builtin_amdgcn_fence(__ATOMIC_ACQUIRE, "");
    const size_t par = (size_t)(call & 1u) * XBYTES;
    {
        char* dst = myws + par + (size_t)b * 8192 + tid * 16;
        asm volatile("global_store_dwordx4 %0, %1, off sc0 sc1\n\ts_nop 1" :: "v"(dst), "v"(xv) : "memory");
    }
    asm volatile("s_waitcnt vmcnt(0)" ::: "memory");
    __builtin_amdgcn_s_barrier();
    if (tid < NR) {
        unsigned* f = (unsigned*)(p.ws[tid] + FLAG_OFF) + (p.rank & 7) * NWG + b;
        __hip_atomic_store(f, call, __ATOMIC_RELAXED, __HIP_MEMORY_SCOPE_SYSTEM);
    }
    if (tid == 64) __hip_atomic_store(cntp, call, __ATOMIC_RELAXED, __HIP_MEMORY_SCOPE_AGENT);

    const int src = (p.rank + (b & 7)) & 7, idx = b >> 3, mt = idx & 1, nt = idx >> 1;
    const unsigned lds0 = (unsigned)(uintptr_t)(LAS char*)smem;

    unsigned wvoff;
    { const int r = tid >> 3, c = tid & 7, f = ((r >> 1) & 3) | (((r >> 5) & 1) << 2); wvoff = (unsigned)(r * NL + ((c ^ f) << 4)); }
    const char* wbase = p.w + nt * 128;
    auto stage_w = [&](int ks) {
        const char* g = wbase + (size_t)ks * (128 * NL);
        const unsigned d = lds0 + (unsigned)(ks & (NSLOT - 1)) * WSLOT + (unsigned)wid * 1024u;
        glds16(wvoff, g, d);
        glds16(wvoff, g + 64 * NL, d + 8192u);
    };
    stage_w(0); stage_w(1); stage_w(2); stage_w(3);

    unsigned* tmo = (unsigned*)(myws + TMO_OFF);
    LAS unsigned* badp = (LAS unsigned*)(smem + NSLOT * WSLOT);
    if (wid == 0) {
        const unsigned long long* fl = (const unsigned long long*)(myws + FLAG_OFF) + (src * NWG + mt * 128) / 2 + lane;
        const unsigned long long t0 = __builtin_amdgcn_s_memrealtime();
        unsigned bad = 0;
        for (;;) {
            const unsigned long long v = __hip_atomic_load(fl, __ATOMIC_RELAXED, __HIP_MEMORY_SCOPE_SYSTEM);
            const bool ok = ((unsigned)v >= call) && ((unsigned)(v >> 32) >= call);
            if (__all(ok)) break;
            if (__hip_atomic_load(tmo, __ATOMIC_RELAXED, __HIP_MEMORY_SCOPE_AGENT) == call ||
                __builtin_amdgcn_s_memrealtime() - t0 > DEADLINE_TICKS) {
                __hip_atomic_store(tmo, call, __ATOMIC_RELAXED, __HIP_MEMORY_SCOPE_AGENT);
                bad = 1; break;
            }
            __builtin_amdgcn_s_sleep(1);
        }
        if (lane == 0) *badp = bad;
        asm volatile("s_waitcnt lgkmcnt(0)" ::: "memory");
    }
    __builtin_amdgcn_s_barrier();
    asm volatile("" ::: "memory");
    const bool bad = __builtin_amdgcn_readfirstlane(*badp) != 0u;

    v4f acc[2][8];
#pragma unroll
    for (int r = 0; r < 2; ++r)
#pragma unroll
        for (int n = 0; n < 8; ++n) acc[r][n] = (v4f){0.f, 0.f, 0.f, 0.f};

    if (!bad) {
        const char* abase = p.ws[src] + par + (size_t)mt * (XBYTES / 2) + (size_t)wid * 4096 + lane * 16;
        v4i ar[4][4];
        auto load_a = [&](v4i (&d)[4], int ks) {
            const char* g = abase + (size_t)ks * 32768;
#pragma unroll
            for (int i = 0; i < 4; ++i) d[i] = *(const v4i*)(g + i * 1024);
        };
        load_a(ar[0], 0); load_a(ar[1], 1); load_a(ar[2], 2); load_a(ar[3], 3);

        unsigned raddr[8];
        { const int g = lane >> 4, u = lane & 15, q = u >> 1, h = u & 1, f = ((q >> 1) & 3) | ((g & 1) << 2);
#pragma unroll
          for (int n = 0; n < 8; ++n) raddr[n] = lds0 + (unsigned)((32 * g + q) * 128 + ((n ^ f) << 4) + h * 8); }

        auto compute = [&](v4i (&a)[4], int slot4) {
            const v8i x0 = {a[0][0], a[0][1], a[0][2], a[0][3], a[1][0], a[1][1], a[1][2], a[1][3]};
            const v8i x1 = {a[2][0], a[2][1], a[2][2], a[2][3], a[3][0], a[3][1], a[3][2], a[3][3]};
#pragma unroll
            for (int n = 0; n < 8; ++n) {
                v2i t[4];
#pragma unroll
                for (int i = 0; i < 4; ++i)
                    t[i] = __builtin_amdgcn_ds_read_tr8_b64_v2i32((LAS v2i*)(raddr[n] + (unsigned)(slot4 * WSLOT + i * 1024)));
                const v8i wf = {t[0][0], t[0][1], t[1][0], t[1][1], t[2][0], t[2][1], t[3][0], t[3][1]};
                acc[0][n] = __builtin_amdgcn_mfma_scale_f32_16x16x128_f8f6f4(wf, x0, acc[0][n], 0, 0, 0, 0, 0, 0);
                acc[1][n] = __builtin_amdgcn_mfma_scale_f32_16x16x128_f8f6f4(wf, x1, acc[1][n], 0, 0, 0, 0, 0, 0);
            }
        };
        for (int k0 = 0; k0 < NSTEP; k0 += 4) {
            const bool more = k0 + 4 < NSTEP;
#pragma unroll
            for (int s = 0; s < 4; ++s) {
                if (more || s == 0) asm volatile("s_waitcnt vmcnt(18)" ::: "memory");
                else if (s == 1) asm volatile("s_waitcnt vmcnt(12)" ::: "memory");
                else if (s == 2) asm volatile("s_waitcnt vmcnt(6)" ::: "memory");
                else asm volatile("s_waitcnt vmcnt(0)" ::: "memory");
                __builtin_amdgcn_s_barrier();
                asm volatile("" ::: "memory");
                compute(ar[s], s);
                asm volatile("" ::: "memory");
                if (more) { stage_w(k0 + s + 4); load_a(ar[s], k0 + s + 4); }
            }
#pragma unroll
            for (int n = 0; n < 8; ++n) raddr[n] ^= 65536u;
        }
    } else {
        asm volatile("s_waitcnt vmcnt(0)" ::: "memory");
    }

    const float scale = p.sx[0] * p.sw[0];
    float* obase = p.out + (size_t)(src * MS + mt * 256 + wid * 32 + (lane & 15)) * NL + nt * 128 + (lane >> 4) * 4;
    const float qnan = __builtin_nanf("");
#pragma unroll
    for (int r = 0; r < 2; ++r)
#pragma unroll
        for (int n = 0; n < 8; ++n) {
            v4f y = acc[r][n] * scale;
#pragma unroll
            for (int e = 0; e < 4; ++e) y[e] = bad ? qnan : (y[e] < 0.f ? 0.f : y[e]);
            *(v4f*)(obase + (size_t)r * 16 * NL + n * 16) = y;
        }
}
}

extern "C" void kernel_launch(int rank, int nranks, ncclComm_t comm,
                              hipStream_t stream,
                              void* const* d_in, const int* in_sizes, int n_in,
                              void* d_out, int out_size,
                              const int* in_dtypes, int out_dtype,
                              void* const* d_out_all,
                              void* const* d_ws_all, size_t ws_bytes) {
    if (nranks != NR || n_in < 4 || ws_bytes < WS_NEED || (unsigned)rank >= (unsigned)NR) {
        fprintf(stderr, "unsupported configuration\n");
        return;
    }
    Params p;
    p.x = (const char*)d_in[0];
    p.w = (const char*)d_in[1];
    p.sx = (const float*)d_in[2];
    p.sw = (const float*)d_in[3];
    p.out = (float*)d_out;
    for (int i = 0; i < NR; ++i) p.ws[i] = (char*)d_ws_all[i];
    p.rank = rank;
    hipLaunchKernelGGL(ag_gemm_kernel, dim3(NWG), dim3(512), 0, stream, p);
    HIP_CHECK(hipGetLastError());
}
```

```cpp
#include <hip/hip_runtime.h>
#include <rccl/rccl.h>
#include <cstdio>
#include <cstdint>

#define HIP_CHECK(expr) do { hipError_t _e = (expr); if (_e != hipSuccess) { \
    fprintf(stderr, "HIP error %s at %s:%d\n", hipGetErrorString(_e), __FILE__, __LINE__); \
    return; } } while (0)

typedef int   v8i __attribute__((ext_vector_type(8)));
typedef int   v4i __attribute__((ext_vector_type(4)));
typedef int   v2i __attribute__((ext_vector_type(2)));
typedef float v4f __attribute__((ext_vector_type(4)));
#define LAS __attribute__((address_space(3)))

#define OSTORE "sc0 sc1"
namespace {
constexpr int NR = 8;
constexpr int KDIM = 4096;
constexpr int NL = 2048;
constexpr int MS = 512;
constexpr int NWG = 256;
constexpr size_t XBYTES = (size_t)MS * KDIM;
constexpr size_t FLAG_OFF = 2 * XBYTES;
constexpr size_t TMO_OFF = FLAG_OFF + 65536;
constexpr size_t WS_NEED = TMO_OFF + 4096;
constexpr unsigned long long DEADLINE_TICKS = 50000000ull;
constexpr int WSLOT = 16384, NSLOT = 8;

struct Params {
    const char* x;
    const char* w;
    const float* sx;
    const float* sw;
    float* out;
    char* myws;
    char* ws[NR];
    int rank;
};

__device__ __forceinline__ void glds16(unsigned voff, const char* sbase, unsigned lds_dst) {
    unsigned keep;
    asm volatile("s_mov_b32 %0, m0\n\ts_mov_b32 m0, %3\n\ts_nop 0\n\tglobal_load_lds_dwordx4 %1, %2\n\ts_mov_b32 m0, %0"
                 : "=&s"(keep) : "v"(voff), "s"(sbase), "s"(lds_dst) : "memory");
}
__device__ __forceinline__ void load_a4(v4i (&d)[4], unsigned voff, const char* sbase) {
    asm volatile("global_load_dwordx4 %0, %4, %5\n\tglobal_load_dwordx4 %1, %4, %5 offset:1024\n\t"
                 "global_load_dwordx4 %2, %4, %5 offset:2048\n\tglobal_load_dwordx4 %3, %4, %5 offset:3072"
                 : "=&v"(d[0]), "=&v"(d[1]), "=&v"(d[2]), "=&v"(d[3]) : "v"(voff), "s"(sbase) : "memory");
}
#define WAIT_A(n, lo, hi) asm volatile("s_waitcnt vmcnt(%2)" : "+v"(lo), "+v"(hi) : "i"(n) : "memory")

#define STAMP(i) do { } while (0)

__global__ void __launch_bounds__(512, 2) ag_gemm_kernel(Params p) {
    STAMP(0);
    __shared__ __attribute__((aligned(16))) char smem[NSLOT * WSLOT + 32768];
    const int tid = threadIdx.x, lane = tid & 63, b = blockIdx.x;
    const int wid = __builtin_amdgcn_readfirstlane(tid >> 6);
    char* myws = p.myws;
    const int rank = p.rank & 7;
    const char* xp = p.x;
    const char* wp = p.w;
    float* outp = p.out;
    asm volatile("" :: "s"(myws), "s"(rank), "s"(xp), "s"(wp), "s"(outp), "s"(p.sx), "s"(p.sw));
    typedef const float __attribute__((address_space(4))) * cfp;
    const float sxv = *(cfp)(uintptr_t)p.sx, swv = *(cfp)(uintptr_t)p.sw;
    const int src = (rank + (b & 7)) & 7, idx = b >> 3, mt = idx & 1, nt = idx >> 1;
    const char* srcws = p.ws[src];

    unsigned* cntp = (unsigned*)(myws + FLAG_OFF) + (rank * 16 + (b >> 4)) * 32;
    unsigned* myflag = nullptr;
    unsigned call5 = 0;
    if (wid < 4) {
        const unsigned cnt_raw = __hip_atomic_load(cntp, __ATOMIC_RELAXED, __HIP_MEMORY_SCOPE_SYSTEM);
        const int cm = b >> 7, v = (b >> 2) & 31, w = (b & 3) * 2 + (wid >> 1), sub = wid & 1;
        const int rb = v < 24 ? sub : (v - 24) >> 2, ks = v < 24 ? v : 24 + ((v & 3) << 1) + sub;
        const int row = cm * 256 + w * 32 + rb * 16 + (lane & 15), kk = ks * 128 + 32 * (lane >> 4);
        const v4i* s = (const v4i*)(xp + (size_t)row * KDIM + kk);
        const v4i xv0 = s[0], xv1 = s[1];
        char* dst = myws + (size_t)(((cnt_raw >> 4) + 1u) & 1u) * XBYTES + (size_t)b * 8192 + wid * 2048 + lane * 16;
        asm volatile("global_store_dwordx4 %0, %1, off sc0 sc1\n\tglobal_store_dwordx4 %0, %2, off offset:1024 sc0 sc1\n\ts_nop 1"
                     :: "v"(dst), "v"(xv0), "v"(xv1) : "memory");
        STAMP(1);
        asm volatile("s_waitcnt vmcnt(0)" ::: "memory");
    } else if (wid == 5) {
        call5 = ((unsigned)__builtin_amdgcn_readfirstlane(__hip_atomic_load(cntp, __ATOMIC_RELAXED, __HIP_MEMORY_SCOPE_SYSTEM)) >> 4) + 1u;
        asm volatile("" :: "s"(call5));
    } else if (wid == 6) {
        myflag = (unsigned*)(p.ws[lane & 7] + FLAG_OFF) + (rank * 16 + (b >> 4)) * 32;
        asm volatile("" :: "v"(myflag));
    }
    const float scale = __builtin_bit_cast(float, __builtin_amdgcn_readfirstlane(__builtin_bit_cast(int, sxv * swv)));
    const unsigned lds0 = (unsigned)(uintptr_t)(LAS char*)smem;
    unsigned wvoff;
    { const int r = tid >> 3, c = tid & 7, f = ((r >> 1) & 3) | (((r >> 5) & 1) << 2); wvoff = (unsigned)(r * NL + ((c ^ f) << 4)); }
    const char* wbase = wp + nt * 128;
    auto stage_w = [&](int ks) {
        const char* g = wbase + (size_t)ks * (128 * NL);
        const unsigned d = lds0 + (unsigned)(ks & (NSLOT - 1)) * WSLOT + (unsigned)wid * 1024u;
        glds16(wvoff, g, d);
        glds16(wvoff, g + 64 * NL, d + 8192u);
    };
    __builtin_amdgcn_s_barrier();
    STAMP(2);
    LAS unsigned* badp = (LAS unsigned*)(smem + NSLOT * WSLOT);
    if (wid == 5) {
        unsigned* tmo = (unsigned*)(myws + TMO_OFF);
        const unsigned* fl = (const unsigned*)(myws + FLAG_OFF) + (src * 16 + mt * 8 + (lane & 7)) * 32;
        const unsigned need = call5 * 16u;
        const unsigned long long t0 = __builtin_amdgcn_s_memrealtime();
        unsigned bad = 0, polls = 0;
        for (;;) {
            const bool ok = __hip_atomic_load(fl, __ATOMIC_RELAXED, __HIP_MEMORY_SCOPE_SYSTEM) >= need;
            if (__all(ok)) break;
            if ((++polls & 63u) == 0u) {
                if (__hip_atomic_load(tmo, __ATOMIC_RELAXED, __HIP_MEMORY_SCOPE_AGENT) == call5 ||
                    __builtin_amdgcn_s_memrealtime() - t0 > DEADLINE_TICKS) {
                    __hip_atomic_store(tmo, call5, __ATOMIC_RELAXED, __HIP_MEMORY_SCOPE_AGENT);
                    bad = 2; break;
                }
            }
            __builtin_amdgcn_s_sleep(1);
        }
        if (lane == 0) *badp = bad | (call5 & 1u);
        asm volatile("s_waitcnt lgkmcnt(0)" ::: "memory");
    } else {
        if (wid == 6 && lane < NR) (void)__hip_atomic_fetch_add(myflag, 1u, __ATOMIC_RELAXED, __HIP_MEMORY_SCOPE_SYSTEM);
        stage_w(0); stage_w(1);
    }
    asm volatile("" :: "s"(scale), "s"(srcws));
    __builtin_amdgcn_s_barrier();
    asm volatile("" ::: "memory");
    const unsigned state = __builtin_amdgcn_readfirstlane(*badp);
    const bool bad = state >= 2u;
    const size_t par = (size_t)(state & 1u) * XBYTES;
    STAMP(3);
    if (wid == 5) { stage_w(0); stage_w(1); }

    v4f acc[2][8];
#pragma unroll
    for (int r = 0; r < 2; ++r)
#pragma unroll
        for (int n = 0; n < 8; ++n) acc[r][n] = (v4f){0.f, 0.f, 0.f, 0.f};

    auto epi_prep = [&](v4f (&c)[8], v4i (&y)[8]) {
        const float qnan = __builtin_nanf("");
        LAS char* ebuf = (LAS char*)smem + NSLOT * WSLOT + wid * 4096;
#pragma unroll
        for (int h = 0; h < 2; ++h) {
#pragma unroll
            for (int n = 0; n < 4; ++n) {
                v4f z = c[h * 4 + n] * scale;
#pragma unroll
                for (int e = 0; e < 4; ++e) z[e] = bad ? qnan : (z[e] < 0.f ? 0.f : z[e]);
                *(LAS v4f*)(ebuf + (lane & 15) * 256 + (((n * 4 + (lane >> 4)) ^ (lane & 7)) << 4)) = z;
            }
#pragma unroll
            for (int i = 0; i < 4; ++i) {
                const int row = 4 * i + (lane >> 4);
                y[h * 4 + i] = *(LAS v4i*)(ebuf + row * 256 + (((lane & 15) ^ (row & 7)) << 4));
            }
        }
    };
    auto epi_store = [&](v4i (&y)[8], int r, int i0, int i1) {
        const char* obase = (const char*)(outp + (size_t)(src * MS + mt * 256 + wid * 32 + r * 16) * NL + nt * 128);
        const unsigned ooff = (unsigned)((lane >> 4) * (NL * 4) + (lane & 15) * 16);
#pragma unroll
        for (int i = 0; i < 4; ++i) if (i >= i0 && i < i1) {
            asm volatile("global_store_dwordx4 %0, %1, %3 " OSTORE "\n\tglobal_store_dwordx4 %0, %2, %3 offset:256 " OSTORE "\n\ts_nop 1"
                         :: "v"(ooff), "v"(y[i]), "v"(y[4 + i]), "s"(obase + i * 4 * NL * 4) : "memory");
        }
        __builtin_amdgcn_sched_barrier(0);
    };
    auto epilogue = [&](v4f (&c)[8], int r) { v4i y[8]; epi_prep(c, y); epi_store(y, r, 0, 4); };

    if (!bad) {
        const char* abase = srcws + par + (size_t)mt * (XBYTES / 2) + (size_t)wid * 4096;
        const unsigned avoff = (unsigned)lane * 16u;
        v4i ar[4][4];
#define LOAD_A(slot, k) load_a4(ar[slot], avoff, abase + (size_t)(k) * 32768)
        LOAD_A(0, 0);
        __builtin_amdgcn_s_barrier();
        stage_w(2); stage_w(3); stage_w(4); stage_w(5);
        stage_w(6); LOAD_A(1, 1);

        unsigned raddr[8];
        { const int g = lane >> 4, u = lane & 15, q = u >> 1, h = u & 1, f = ((q >> 1) & 3) | ((g & 1) << 2);
#pragma unroll
          for (int n = 0; n < 8; ++n) raddr[n] = lds0 + (unsigned)((32 * g + q) * 128 + ((n ^ f) << 4) + h * 8); }

        auto wfrag = [&](int n, int slot4) {
            v2i t[4];
#pragma unroll
            for (int i = 0; i < 4; ++i)
                t[i] = __builtin_amdgcn_ds_read_tr8_b64_v2i32((LAS v2i*)(uintptr_t)(raddr[n] + (unsigned)(slot4 * WSLOT + i * 1024)));
            return (v8i){t[0][0], t[0][1], t[1][0], t[1][1], t[2][0], t[2][1], t[3][0], t[3][1]};
        };
        auto compute = [&](v4i (&a)[4], int slot4) {
            const v8i x0 = {a[0][0], a[0][1], a[0][2], a[0][3], a[1][0], a[1][1], a[1][2], a[1][3]};
            const v8i x1 = {a[2][0], a[2][1], a[2][2], a[2][3], a[3][0], a[3][1], a[3][2], a[3][3]};
            v8i wf[8];
#pragma unroll
            for (int n = 0; n < 8; ++n) wf[n] = wfrag(n, slot4);
#pragma unroll
            for (int n = 0; n < 8; ++n) {
                acc[0][n] = __builtin_amdgcn_mfma_scale_f32_16x16x128_f8f6f4(wf[n], x0, acc[0][n], 0, 0, 0, 0, 0, 0);
                acc[1][n] = __builtin_amdgcn_mfma_scale_f32_16x16x128_f8f6f4(wf[n], x1, acc[1][n], 0, 0, 0, 0, 0, 0);
            }
            __builtin_amdgcn_sched_group_barrier(0x100, 12, 0);
#pragma unroll
            for (int n = 0; n < 5; ++n) {
                __builtin_amdgcn_sched_group_barrier(0x8, 2, 0);
                __builtin_amdgcn_sched_group_barrier(0x100, 4, 0);
            }
            __builtin_amdgcn_sched_group_barrier(0x8, 6, 0);
            __builtin_amdgcn_sched_barrier(0);
        };
        auto compute1 = [&](v4f (&c)[8], const v4i& lo, const v4i& hi, int slot4) {
            const v8i x0 = {lo[0], lo[1], lo[2], lo[3], hi[0], hi[1], hi[2], hi[3]};
            v8i wf[8];
#pragma unroll
            for (int n = 0; n < 8; ++n) wf[n] = wfrag(n, slot4);
#pragma unroll
            for (int n = 0; n < 8; ++n) c[n] = __builtin_amdgcn_mfma_scale_f32_16x16x128_f8f6f4(wf[n], x0, c[n], 0, 0, 0, 0, 0, 0);
            __builtin_amdgcn_sched_group_barrier(0x100, 12, 0);
#pragma unroll
            for (int n = 0; n < 5; ++n) {
                __builtin_amdgcn_sched_group_barrier(0x8, 1, 0);
                __builtin_amdgcn_sched_group_barrier(0x100, 4, 0);
            }
            __builtin_amdgcn_sched_group_barrier(0x8, 3, 0);
            __builtin_amdgcn_sched_barrier(0);
        };
#define WAIT_A4(n, a) asm volatile("s_waitcnt vmcnt(%4)" : "+v"(a[0]), "+v"(a[1]), "+v"(a[2]), "+v"(a[3]) : "i"(n) : "memory")
        asm volatile("s_waitcnt vmcnt(18)" ::: "memory");
#define STAMPK(i) do { } while (0)
        for (int k0 = 0; k0 < 24; k0 += 4) {
#pragma unroll
            for (int s = 0; s < 4; ++s) {
                STAMPK(4);
                __builtin_amdgcn_s_barrier();
                STAMPK(5);
                WAIT_A4(6, ar[s]);
                STAMPK(6);
                compute(ar[s], s);
                STAMPK(7);
                stage_w(k0 + s + 7);
                STAMPK(8);
                LOAD_A((s + 2) & 3, k0 + s + 2);
                STAMPK(9);
            }
#pragma unroll
            for (int n = 0; n < 8; ++n) raddr[n] ^= 65536u;
        }
        STAMP(10);
#define SA 1
#define SB 2
        constexpr int SC = 4 - SA - SB;
        __builtin_amdgcn_s_barrier(); WAIT_A4(6, ar[0]);
        compute1(acc[0], ar[0][0], ar[0][1], 0); compute1(acc[0], ar[0][2], ar[0][3], 1);
        stage_w(31); LOAD_A(2, 26);
        __builtin_amdgcn_s_barrier(); WAIT_A4(6, ar[1]);
        compute1(acc[0], ar[1][0], ar[1][1], 2); compute1(acc[0], ar[1][2], ar[1][3], 3);
        LOAD_A(3, 27);
#pragma unroll
        for (int n = 0; n < 8; ++n) raddr[n] ^= 65536u;
        __builtin_amdgcn_s_barrier(); WAIT_A4(4, ar[2]);
        compute1(acc[0], ar[2][0], ar[2][1], 0); compute1(acc[0], ar[2][2], ar[2][3], 1);
        LOAD_A(0, 28);
        __builtin_amdgcn_s_barrier(); WAIT_A4(4, ar[3]);
        compute1(acc[0], ar[3][0], ar[3][1], 2); compute1(acc[0], ar[3][2], ar[3][3], 3);
        LOAD_A(1, 29);
        v4i y0[8];
        epi_prep(acc[0], y0);
        epi_store(y0, 0, 0, SA);
        STAMP(11);
#pragma unroll
        for (int n = 0; n < 8; ++n) raddr[n] ^= 65536u;
        WAIT_A4(4 + 2 * SA, ar[0]);
        compute1(acc[1], ar[0][0], ar[0][1], 0); compute1(acc[1], ar[0][2], ar[0][3], 1);
        LOAD_A(2, 30); epi_store(y0, 0, SA, SA + SB);
        WAIT_A4(4 + 2 * SA + 2 * SB, ar[1]);
        compute1(acc[1], ar[1][0], ar[1][1], 2); compute1(acc[1], ar[1][2], ar[1][3], 3);
        LOAD_A(3, 31); epi_store(y0, 0, SA + SB, 4);
#pragma unroll
        for (int n = 0; n < 8; ++n) raddr[n] ^= 65536u;
        WAIT_A(6 + 2 * SB + 2 * SC, ar[2][0], ar[2][1]); compute1(acc[1], ar[2][0], ar[2][1], 0);
        WAIT_A(4 + 2 * SB + 2 * SC, ar[2][2], ar[2][3]); compute1(acc[1], ar[2][2], ar[2][3], 1);
        WAIT_A(2 + 2 * SC, ar[3][0], ar[3][1]);  compute1(acc[1], ar[3][0], ar[3][1], 2);
        WAIT_A(2 * SC, ar[3][2], ar[3][3]);      compute1(acc[1], ar[3][2], ar[3][3], 3);
        STAMP(12);
        epilogue(acc[1], 1);
    } else {
        asm volatile("s_waitcnt vmcnt(0)" ::: "memory");
        __builtin_amdgcn_s_barrier();
        epilogue(acc[0], 0);
        epilogue(acc[1], 1);
    }
}
}

extern "C" void kernel_launch(int rank, int nranks, ncclComm_t comm,
                              hipStream_t stream,
                              void* const* d_in, const int* in_sizes, int n_in,
                              void* d_out, int out_size,
                              const int* in_dtypes, int out_dtype,
                              void* const* d_out_all,
                              void* const* d_ws_all, size_t ws_bytes) {
    if (nranks != NR || n_in < 4 || ws_bytes < WS_NEED || (unsigned)rank >= (unsigned)NR) {
        fprintf(stderr, "unsupported configuration\n");
        return;
    }
    Params p;
    p.x = (const char*)d_in[0];
    p.w = (const char*)d_in[1];
    p.sx = (const float*)d_in[2];
    p.sw = (const float*)d_in[3];
    p.out = (float*)d_out;
    for (int i = 0; i < NR; ++i) p.ws[i] = (char*)d_ws_all[i];
    p.rank = rank;
    p.myws = (char*)d_ws_all[rank];
    hipLaunchKernelGGL(ag_gemm_kernel, dim3(NWG), dim3(512), 0, stream, p);
    HIP_CHECK(hipGetLastError());
}
```

```cpp
#include <hip/hip_runtime.h>
#include <rccl/rccl.h>
#include <cstdio>
#include <cstdint>

#define HIP_CHECK(expr) do { hipError_t _e = (expr); if (_e != hipSuccess) { \
    fprintf(stderr, "HIP error %s at %s:%d\n", hipGetErrorString(_e), __FILE__, __LINE__); \
    return; } } while (0)

typedef int   v8i __attribute__((ext_vector_type(8)));
typedef int   v4i __attribute__((ext_vector_type(4)));
typedef int   v2i __attribute__((ext_vector_type(2)));
typedef float v4f __attribute__((ext_vector_type(4)));
#define LAS __attribute__((address_space(3)))
#define GAS __attribute__((address_space(1)))

#define OSTORE "sc0 sc1"
#define PSLEEP 16
namespace {
constexpr int NR = 8;
constexpr int KDIM = 4096;
constexpr int NL = 2048;
constexpr int MS = 512;
constexpr int NWG = 256;
constexpr size_t XBYTES = (size_t)MS * KDIM;
constexpr size_t FLAG_OFF = 2 * XBYTES;
constexpr size_t PFLAG_OFF = FLAG_OFF + 32768;
constexpr size_t TMO_OFF = FLAG_OFF + 65536;
constexpr size_t XOFF_OFF = TMO_OFF + 4096;
constexpr size_t CNT_OFF = FLAG_OFF + 131072;
constexpr size_t RECV_OFF = 16 * XBYTES;
constexpr size_t WS_NEED = RECV_OFF + 16 * XBYTES;
#define PK0V 8
constexpr int PK0 = PK0V;
constexpr unsigned long long DEADLINE_TICKS = 50000000ull;
constexpr int WSLOT = 16384, NSLOT = 8, EBUF = NSLOT * WSLOT;

struct Params {
    const char* x;
    const char* w;
    const float* sx;
    const float* sw;
    float* out;
    char* myws;
    char* ws[NR];
    int rank;
    unsigned srcmap;
};

__device__ __forceinline__ void glds16(unsigned voff, const char* sbase, unsigned lds_dst) {
    unsigned keep;
    asm volatile("s_mov_b32 %0, m0\n\ts_mov_b32 m0, %3\n\ts_nop 0\n\tglobal_load_lds_dwordx4 %1, %2\n\ts_mov_b32 m0, %0"
                 : "=&s"(keep) : "v"(voff), "s"(sbase), "s"(lds_dst) : "memory");
}
__device__ __forceinline__ void load_a4(v4i (&d)[4], unsigned voff, const char* sbase) {
    asm volatile("global_load_dwordx4 %0, %4, %5\n\tglobal_load_dwordx4 %1, %4, %5 offset:1024\n\t"
                 "global_load_dwordx4 %2, %4, %5 offset:2048\n\tglobal_load_dwordx4 %3, %4, %5 offset:3072"
                 : "=&v"(d[0]), "=&v"(d[1]), "=&v"(d[2]), "=&v"(d[3]) : "v"(voff), "s"(sbase) : "memory");
}
#define WAIT_A(n, lo, hi) asm volatile("s_waitcnt vmcnt(%2)" : "+v"(lo), "+v"(hi) : "i"(n) : "memory")
#define WAIT_A4(n, a) asm volatile("s_waitcnt vmcnt(%4)" : "+v"(a[0]), "+v"(a[1]), "+v"(a[2]), "+v"(a[3]) : "i"(n) : "memory")

#define STAMP(i) do { } while (0)

__global__ void __launch_bounds__(512, 2) ag_gemm_kernel(Params p) {
    STAMP(0);
    __shared__ __attribute__((aligned(16))) char smem[EBUF + 32768];
    const int tid = threadIdx.x, lane = tid & 63, b = blockIdx.x;
    const int wid = __builtin_amdgcn_readfirstlane(tid >> 6);
    char* myws = p.myws;
    const int rank = p.rank & 7;
    const char* xp = p.x;
    const char* wp = p.w;
    float* outp = p.out;
    const unsigned srcmap = p.srcmap;
    asm volatile("" :: "s"(myws), "s"(rank), "s"(xp), "s"(wp), "s"(outp), "s"(p.sx), "s"(p.sw), "s"(srcmap));
    typedef const float __attribute__((address_space(4))) * cfp;
    const float sxv = *(cfp)(uintptr_t)p.sx, swv = *(cfp)(uintptr_t)p.sw;
    const int idx = b >> 3, mt = idx & 1, nt = idx >> 1;
    unsigned long long wsl = 0;

    const int cb = b;
    const int shard = ((cb >> 2) & 1) * 8 + (cb >> 5);
    unsigned* cntp = (unsigned*)(myws + CNT_OFF) + b * 32;
    unsigned call5 = 0;
    v4i pv = {0, 0, 0, 0};
    unsigned poff = 0;
    const int pf = (b >> 3) * 4 + (b & 3), phalf = (b >> 2) & 1;
    auto push_one = [&](int i) {
        const int sl = phalf * 4 + ((i + pf) & 3);
        if (sl != 0) asm volatile("global_store_dwordx4 %0, %1, %2 sc0 sc1\n\ts_nop 1" :: "v"(poff), "v"(pv), "s"(p.ws[sl]) : "memory");
    };
    if (wid < 4) {
        const unsigned cnt_raw = __hip_atomic_load(cntp, __ATOMIC_RELAXED, __HIP_MEMORY_SCOPE_AGENT);
        const int v = cb >> 3, cm = (cb >> 2) & 1, sub = wid & 1;
        const int rb = v < 24 ? sub : (v - 24) >> 2, ks = v < 24 ? v : 24 + ((v & 3) << 1) + sub;
        const int row = cm * 256 + (cb & 3) * 64 + (wid >> 1) * 32 + rb * 16 + (lane & 15), kk = ks * 128 + 32 * (lane >> 4);
        const v4i* s = (const v4i*)(xp + (size_t)row * KDIM + kk);
        const v4i xv0 = s[0], xv1 = s[1];
        char* dst = myws + (size_t)((cnt_raw + 1u) & 1u) * XBYTES + ((cm * 32 + v) * 4 + (cb & 3)) * 8192 + wid * 2048 + lane * 16;
        asm volatile("global_store_dwordx4 %0, %1, off sc0 sc1\n\tglobal_store_dwordx4 %0, %2, off offset:1024 sc0 sc1\n\ts_nop 1"
                     :: "v"(dst), "v"(xv0), "v"(xv1) : "memory");
        STAMP(1);
        asm volatile("s_waitcnt vmcnt(0)" ::: "memory");
    } else if (wid == 5) {
        call5 = (unsigned)__builtin_amdgcn_readfirstlane(__hip_atomic_load(cntp, __ATOMIC_RELAXED, __HIP_MEMORY_SCOPE_AGENT)) + 1u;
        if (b == 0)
            __hip_atomic_store((unsigned*)(myws + XOFF_OFF) + lane * 32, call5 * 8u + (__builtin_amdgcn_s_getreg((31 << 11) | 20) & 7u),
                               __ATOMIC_RELAXED, __HIP_MEMORY_SCOPE_AGENT);
    } else if (wid == 6) {
        wsl = (unsigned long long)(p.ws[lane & 7] + FLAG_OFF + (rank * 16 + shard) * 128);
        asm volatile("" :: "v"(wsl));
    } else if (wid == 7) {
        wsl = (unsigned long long)(p.ws[phalf * 4 + (lane & 3)] + PFLAG_OFF + (rank * 8 + (pf >> 4)) * 128);
        const unsigned cnt_raw = __hip_atomic_load(cntp, __ATOMIC_RELAXED, __HIP_MEMORY_SCOPE_AGENT);
        const int pm = pf >> 6, pk = PK0 + ((pf >> 2) & 15), q = pf & 3;
        const int prb = pk < 24 ? (q >> 1) : (pk - 24) >> 2, pks = pk < 24 ? pk : 24 + ((pk & 3) << 1) + (q >> 1);
        const int row = pm * 256 + 224 + prb * 16 + (lane & 15), kk = pks * 128 + 32 * (lane >> 4) + 16 * (q & 1);
        pv = *(const v4i*)(xp + (size_t)row * KDIM + kk);
        poff = (unsigned)RECV_OFF + (((cnt_raw + 1u) & 1u) * 8u + (unsigned)rank) * (unsigned)XBYTES +
               (unsigned)(pm * (int)(XBYTES / 2) + pk * 32768 + 7 * 4096 + q * 1024 + lane * 16);
        asm volatile("" : "+v"(pv), "+v"(poff), "+v"(wsl));
        push_one(0);
    }
    const float scale = __builtin_bit_cast(float, __builtin_amdgcn_readfirstlane(__builtin_bit_cast(int, sxv * swv)));
    const unsigned lds0 = (unsigned)(uintptr_t)(LAS char*)smem;
    auto w_voff = [&](int v) {
        const int r = v * 8 + (lane >> 3), c = lane & 7, f = ((r >> 1) & 3) | (((r >> 5) & 1) << 2);
        return (unsigned)(r * NL + ((c ^ f) << 4));
    };
    const unsigned wvoff = w_voff(wid);
    const char* wbase = wp + nt * 128;
    auto stage_wv = [&](int ks, unsigned voff, int v) {
        const char* g = wbase + (size_t)ks * (128 * NL);
        const unsigned d = lds0 + (unsigned)(ks & (NSLOT - 1)) * WSLOT + (unsigned)v * 1024u;
        glds16(voff, g, d);
        glds16(voff, g + 64 * NL, d + 8192u);
    };
    auto stage_w = [&](int ks) { stage_wv(ks, wvoff, wid); };
    __builtin_amdgcn_s_barrier();
    STAMP(2);
    LAS unsigned* badp = (LAS unsigned*)(smem + EBUF + 5 * 4096);
    unsigned* tmo = (unsigned*)(myws + TMO_OFF);
    if (wid == 5) {
        unsigned xoff = (__builtin_amdgcn_s_getreg((31 << 11) | 20) - (unsigned)b) & 7u;
        const unsigned need = call5 * 16u;
        const unsigned long long t0 = __builtin_amdgcn_s_memrealtime();
        unsigned bad = 0, polls = 0, src5;
        unsigned long long sws;
        for (;;) {
            src5 = (srcmap >> (((b + xoff) & 7u) * 4u)) & 7u;
            sws = (unsigned long long)p.ws[(b + xoff) & 7u];
            unsigned v, xw;
            {
                const unsigned* fl = (const unsigned*)(myws + (lane == 0 ? FLAG_OFF + (src5 * 16 + mt * 8) * 128 : XOFF_OFF + (b >> 2) * 128));
                const unsigned t = __hip_atomic_load(fl, __ATOMIC_RELAXED, __HIP_MEMORY_SCOPE_SYSTEM);
                v = (unsigned)__builtin_amdgcn_readlane((int)t, 0);
                xw = (unsigned)__builtin_amdgcn_readlane((int)t, 1);
                asm volatile("" : "+s"(sws));
            }
            if ((xw >> 3) == (call5 & 0x1fffffffu)) {
                if ((xw & 7u) != xoff) { xoff = xw & 7u; continue; }
                if (v >= need) break;
            }
            if ((++polls & 255u) == 0u) {
                if (__hip_atomic_load(tmo, __ATOMIC_RELAXED, __HIP_MEMORY_SCOPE_AGENT) == call5 ||
                    __builtin_amdgcn_s_memrealtime() - t0 > DEADLINE_TICKS) {
                    __hip_atomic_store(tmo, call5, __ATOMIC_RELAXED, __HIP_MEMORY_SCOPE_AGENT);
                    bad = 8; break;
                }
            }
#define POLLSLEEP 1
            __builtin_amdgcn_s_sleep(POLLSLEEP);
        }
        if (lane == 0) {
            __hip_atomic_store(cntp, call5, __ATOMIC_RELAXED, __HIP_MEMORY_SCOPE_AGENT);
            badp[0] = bad | src5; badp[1] = call5; badp[2] = (unsigned)sws; badp[3] = (unsigned)(sws >> 32);
        }
        asm volatile("s_waitcnt lgkmcnt(0)" ::: "memory");
    } else {
        if (wid == 6 && lane < NR)
            (void)__hip_atomic_fetch_add((GAS unsigned*)wsl, 1u, __ATOMIC_RELAXED, __HIP_MEMORY_SCOPE_SYSTEM);
        if (wid == 7) push_one(1);
        stage_w(0);
        if (wid == 7) { __builtin_amdgcn_s_sleep(PSLEEP); push_one(2); __builtin_amdgcn_s_sleep(PSLEEP); push_one(3); }
    }
    asm volatile("" :: "s"(scale));
    __builtin_amdgcn_s_barrier();
    asm volatile("" ::: "memory");
    const unsigned state = __builtin_amdgcn_readfirstlane(badp[0]);
    const unsigned call = __builtin_amdgcn_readfirstlane(badp[1]);
    bool bad = state >= 8u;
    const int src = (int)(state & 7u);
    const char* srcws = (const char*)(((unsigned long long)(unsigned)__builtin_amdgcn_readfirstlane(badp[3]) << 32) |
                                      (unsigned)__builtin_amdgcn_readfirstlane(badp[2]));
    STAMP(3);
    if (wid == 5) stage_w(0);
    auto push_signal = [&]() {
        if (lane < 4 && phalf * 4 + lane != 0)
            (void)__hip_atomic_fetch_add((GAS unsigned*)wsl, 1u, __ATOMIC_RELAXED, __HIP_MEMORY_SCOPE_SYSTEM);
    };

    v4f acc[2][8];
#pragma unroll
    for (int r = 0; r < 2; ++r)
#pragma unroll
        for (int n = 0; n < 8; ++n) acc[r][n] = (v4f){0.f, 0.f, 0.f, 0.f};

    auto epi_prep = [&](v4f (&c)[8], v4i (&y)[8], int h) {
        const float sc = bad ? __builtin_nanf("") : scale;
        LAS char* ebuf = (LAS char*)smem + EBUF + wid * 4096;
#pragma unroll
        for (int n = 0; n < 4; ++n) {
            const v4f z = __builtin_elementwise_maximum(c[h * 4 + n] * sc, (v4f){0.f, 0.f, 0.f, 0.f});
            *(LAS v4f*)(ebuf + (lane & 15) * 256 + (((n * 4 + (lane >> 4)) ^ (lane & 7)) << 4)) = z;
        }
#pragma unroll
        for (int i = 0; i < 4; ++i) {
            const int row = 4 * i + (lane >> 4);
            y[h * 4 + i] = *(LAS v4i*)(ebuf + row * 256 + (((lane & 15) ^ (row & 7)) << 4));
        }
    };
    auto epi_store = [&](v4i (&y)[8], int r, int j0, int j1) {
        const char* obase = (const char*)(outp + (size_t)(src * MS + mt * 256 + wid * 32 + r * 16) * NL + nt * 128);
        const unsigned ooff = (unsigned)((lane >> 4) * (NL * 4) + (lane & 15) * 16);
#pragma unroll
        for (int j = 0; j < 8; ++j) if (j >= j0 && j < j1)
            asm volatile("global_store_dwordx4 %0, %1, %2 offset:%3 " OSTORE "\n\ts_nop 1"
                         :: "v"(ooff), "v"(y[j]), "s"(obase + (j & 3) * 4 * NL * 4), "i"((j >> 2) * 256) : "memory");
        __builtin_amdgcn_sched_barrier(0);
    };
    auto epilogue = [&](v4f (&c)[8], int r) { v4i y[8]; epi_prep(c, y, 0); epi_store(y, r, 0, 4); epi_prep(c, y, 1); epi_store(y, r, 4, 8); };

    if (!bad) {
        const size_t aoff = (size_t)mt * (XBYTES / 2) + (size_t)wid * 4096;
        const char* abase = srcws + (size_t)(call & 1u) * XBYTES + aoff;
        const bool pushed = wid == 7 && src != rank;
        const char* pbase = pushed ? myws + RECV_OFF + (size_t)((call & 1u) * 8u + (unsigned)src) * XBYTES + aoff : abase;
        const unsigned avoff = (unsigned)lane * 16u;
        v4i ar[4][4];
#define LOAD_AS(slot, k) load_a4(ar[slot], avoff, ((unsigned)((k) - PK0) < 16u ? pbase : abase) + (size_t)(k) * 32768)
#define LOAD_A(k) LOAD_AS((k) & 3, k)
        LOAD_A(0);
        stage_w(1);
        __builtin_amdgcn_s_barrier();
        stage_w(2); LOAD_A(1);
        int wn = 3;

        unsigned raddr[8];
        { const int g = lane >> 4, u = lane & 15, q = u >> 1, h = u & 1, f = ((q >> 1) & 3) | ((g & 1) << 2);
#pragma unroll
          for (int n = 0; n < 8; ++n) raddr[n] = lds0 + (unsigned)((32 * g + q) * 128 + ((n ^ f) << 4) + h * 8); }
        auto flip = [&]() {
#pragma unroll
            for (int n = 0; n < 8; ++n) raddr[n] ^= 65536u;
        };
        auto wfrag = [&](int n, unsigned off) {
            v2i t[4];
#pragma unroll
            for (int i = 0; i < 4; ++i)
                t[i] = __builtin_amdgcn_ds_read_tr8_b64_v2i32((LAS v2i*)(uintptr_t)(raddr[n] + off + (unsigned)(i * 1024)));
            return (v8i){t[0][0], t[0][1], t[1][0], t[1][1], t[2][0], t[2][1], t[3][0], t[3][1]};
        };
        auto compute = [&](v4i (&a)[4], int slot4) {
            const v8i x0 = {a[0][0], a[0][1], a[0][2], a[0][3], a[1][0], a[1][1], a[1][2], a[1][3]};
            const v8i x1 = {a[2][0], a[2][1], a[2][2], a[2][3], a[3][0], a[3][1], a[3][2], a[3][3]};
            v8i wf[8];
#pragma unroll
            for (int n = 0; n < 8; ++n) wf[n] = wfrag(n, slot4 * WSLOT);
#pragma unroll
            for (int n = 0; n < 8; ++n) {
                acc[0][n] = __builtin_amdgcn_mfma_scale_f32_16x16x128_f8f6f4(wf[n], x0, acc[0][n], 0, 0, 0, 0, 0, 0);
                acc[1][n] = __builtin_amdgcn_mfma_scale_f32_16x16x128_f8f6f4(wf[n], x1, acc[1][n], 0, 0, 0, 0, 0, 0);
            }
            __builtin_amdgcn_sched_group_barrier(0x100, 12, 0);
#pragma unroll
            for (int n = 0; n < 5; ++n) {
                __builtin_amdgcn_sched_group_barrier(0x8, 2, 0);
                __builtin_amdgcn_sched_group_barrier(0x100, 4, 0);
            }
            __builtin_amdgcn_sched_group_barrier(0x8, 6, 0);
            __builtin_amdgcn_sched_barrier(0);
        };
        auto compute1 = [&](v4f (&c)[8], const v4i& lo, const v4i& hi, int slot4) {
            const v8i x0 = {lo[0], lo[1], lo[2], lo[3], hi[0], hi[1], hi[2], hi[3]};
            v8i wf[8];
#pragma unroll
            for (int n = 0; n < 8; ++n) wf[n] = wfrag(n, slot4 * WSLOT);
#pragma unroll
            for (int n = 0; n < 8; ++n) c[n] = __builtin_amdgcn_mfma_scale_f32_16x16x128_f8f6f4(wf[n], x0, c[n], 0, 0, 0, 0, 0, 0);
            __builtin_amdgcn_sched_group_barrier(0x100, 12, 0);
#pragma unroll
            for (int n = 0; n < 5; ++n) {
                __builtin_amdgcn_sched_group_barrier(0x8, 1, 0);
                __builtin_amdgcn_sched_group_barrier(0x100, 4, 0);
            }
            __builtin_amdgcn_sched_group_barrier(0x8, 3, 0);
            __builtin_amdgcn_sched_barrier(0);
        };
        auto ctr_fetch = [&](const char* base) {
            unsigned keep;
            asm volatile("s_mov_b32 %0, m0\n\ts_mov_b32 m0, %3\n\ts_nop 0\n\tglobal_load_lds_dword %1, %2 sc0 sc1\n\ts_mov_b32 m0, %0"
                         : "=&s"(keep) : "v"((unsigned)(lane & 7) * 128u), "s"(base), "s"(lds0 + EBUF + wid * 4096) : "memory");
        };
        auto ctr_check = [&](const char* base) {
            const unsigned need = call * 16u;
            if (__all(*(LAS unsigned*)(smem + EBUF + wid * 4096 + lane * 4) >= need)) return false;
            const unsigned* fl = (const unsigned*)base + (lane & 7) * 32;
            const unsigned long long t0 = __builtin_amdgcn_s_memrealtime();
            while (!__all(__hip_atomic_load(fl, __ATOMIC_RELAXED, __HIP_MEMORY_SCOPE_SYSTEM) >= need)) {
                if (__hip_atomic_load(tmo, __ATOMIC_RELAXED, __HIP_MEMORY_SCOPE_AGENT) == call ||
                    __builtin_amdgcn_s_memrealtime() - t0 > DEADLINE_TICKS) {
                    __hip_atomic_store(tmo, call, __ATOMIC_RELAXED, __HIP_MEMORY_SCOPE_AGENT);
                    return true;
                }
                __builtin_amdgcn_s_sleep(2);
            }
            return false;
        };
        const char* cbase = wid == 4 ? myws + FLAG_OFF + (src * 16 + mt * 8) * 128 : myws + PFLAG_OFF + src * 1024;
        const int ck0 = wid == 4 ? 0 : (pushed ? 4 : -1);
        if (wid == 4) ctr_fetch(cbase);
        asm volatile("s_waitcnt vmcnt(12)" ::: "memory");
        for (int k0 = 0; k0 < 24; k0 += 4) {
#pragma unroll
            for (int s = 0; s < 4; ++s) {
                __builtin_amdgcn_s_barrier();
                WAIT_A4(6, ar[s]);
                compute(ar[s], s);
                if (s == 0 && k0 == 0 && wid == 7) push_signal();
                if (s == 1 && k0 == ck0) {
                    if (ctr_check(cbase)) { if (lane == 0) badp[0] = 8u; }
                    asm volatile("s_waitcnt lgkmcnt(0)" ::: "memory");
                }
#define XW0 4
                if (s == 1 && k0 >= XW0 && k0 < XW0 + 16) stage_w(wn++);
                stage_w(wn++);
                LOAD_AS((s + 2) & 3, k0 + s + 2);
                if (s == 3 && k0 == 0 && pushed) ctr_fetch(cbase);
            }
            flip();
        }
        STAMP(10);
        bad = (unsigned)__builtin_amdgcn_readfirstlane(badp[0]) >= 8u;
#define SA 2
#define SB 4
        constexpr int SC = 8 - SA - SB;
        __builtin_amdgcn_s_barrier(); WAIT_A4(6, ar[0]);
        compute1(acc[0], ar[0][0], ar[0][1], 0); compute1(acc[0], ar[0][2], ar[0][3], 1);
        stage_w(31); LOAD_A(26);
        __builtin_amdgcn_s_barrier(); WAIT_A4(6, ar[1]);
        compute1(acc[0], ar[1][0], ar[1][1], 2); compute1(acc[0], ar[1][2], ar[1][3], 3);
        LOAD_A(27);
        flip();
        __builtin_amdgcn_s_barrier(); WAIT_A4(4, ar[2]);
        compute1(acc[0], ar[2][0], ar[2][1], 0); compute1(acc[0], ar[2][2], ar[2][3], 1);
        LOAD_A(28);
        __builtin_amdgcn_s_barrier(); WAIT_A4(4, ar[3]);
        compute1(acc[0], ar[3][0], ar[3][1], 2); compute1(acc[0], ar[3][2], ar[3][3], 3);
        LOAD_A(29);
        v4i y0[8];
        epi_prep(acc[0], y0, 0); epi_prep(acc[0], y0, 1);
        epi_store(y0, 0, 0, SA);
        flip();
        WAIT_A4(4 + SA, ar[0]);
        compute1(acc[1], ar[0][0], ar[0][1], 0); compute1(acc[1], ar[0][2], ar[0][3], 1);
        LOAD_A(30); epi_store(y0, 0, SA, SA + SB);
        WAIT_A4(4 + SA + SB, ar[1]);
        compute1(acc[1], ar[1][0], ar[1][1], 2); compute1(acc[1], ar[1][2], ar[1][3], 3);
        LOAD_A(31); epi_store(y0, 0, SA + SB, 8);
        flip();
        WAIT_A(6 + SB + SC, ar[2][0], ar[2][1]); compute1(acc[1], ar[2][0], ar[2][1], 0);
        WAIT_A(4 + SB + SC, ar[2][2], ar[2][3]); compute1(acc[1], ar[2][2], ar[2][3], 1);
        WAIT_A(2 + SC, ar[3][0], ar[3][1]);  compute1(acc[1], ar[3][0], ar[3][1], 2);
        WAIT_A(SC, ar[3][2], ar[3][3]);      compute1(acc[1], ar[3][2], ar[3][3], 3);
        STAMP(12);
        epilogue(acc[1], 1);
    } else {
        asm volatile("s_waitcnt vmcnt(0)" ::: "memory");
        if (wid == 7) push_signal();
        __builtin_amdgcn_s_barrier();
        epilogue(acc[0], 0);
        epilogue(acc[1], 1);
    }
}
}

extern "C" void kernel_launch(int rank, int nranks, ncclComm_t comm,
                              hipStream_t stream,
                              void* const* d_in, const int* in_sizes, int n_in,
                              void* d_out, int out_size,
                              const int* in_dtypes, int out_dtype,
                              void* const* d_out_all,
                              void* const* d_ws_all, size_t ws_bytes) {
    if (nranks != NR || n_in < 4 || ws_bytes < WS_NEED || (unsigned)rank >= (unsigned)NR) {
        fprintf(stderr, "unsupported configuration\n");
        return;
    }
    Params p;
    p.x = (const char*)d_in[0];
    p.w = (const char*)d_in[1];
    p.sx = (const float*)d_in[2];
    p.sw = (const float*)d_in[3];
    p.out = (float*)d_out;
    p.rank = rank;
    p.myws = (char*)d_ws_all[rank];
    {
        const unsigned setB = 0xB2u;
        const unsigned mine = (setB >> rank) & 1u;
        unsigned map = (unsigned)rank, lo = 1, hi = 4;
        for (int i = 1; i < NR; ++i) {
            const unsigned r = (unsigned)(rank + i) & 7u;
            if (((setB >> r) & 1u) == mine) map |= r << (4 * lo++); else map |= r << (4 * hi++);
        }
        p.srcmap = map;
        for (int i = 0; i < NR; ++i) p.ws[i] = (char*)d_ws_all[(map >> (4 * i)) & 7u];
    }
    hipLaunchKernelGGL(ag_gemm_kernel, dim3(NWG), dim3(512), 0, stream, p);
    HIP_CHECK(hipGetLastError());
}
```

```cpp
#include <hip/hip_runtime.h>
#include <rccl/rccl.h>
#include <cstdio>
#include <cstdint>

#define HIP_CHECK(expr) do { hipError_t _e = (expr); if (_e != hipSuccess) { \
    fprintf(stderr, "HIP error %s at %s:%d\n", hipGetErrorString(_e), __FILE__, __LINE__); \
    return; } } while (0)

typedef int   v8i __attribute__((ext_vector_type(8)));
typedef int   v4i __attribute__((ext_vector_type(4)));
typedef int   v2i __attribute__((ext_vector_type(2)));
typedef float v4f __attribute__((ext_vector_type(4)));
#define LAS __attribute__((address_space(3)))
#define GAS __attribute__((address_space(1)))

#define OSTORE "sc0 sc1"
#define SPLITISSUE 0xaa
#define URG 1
#define POLLSLEEP 1
#define LAY 0x10432
#define LAYV LAY
#define PREV SPLITISSUE
namespace {
constexpr int NR = 8;
constexpr int KDIM = 4096;
constexpr int NL = 2048;
constexpr int MS = 512;
constexpr int NWG = 256;
constexpr size_t XBYTES = (size_t)MS * KDIM;
constexpr size_t FLAG_OFF = 0;
constexpr size_t TMO_OFF = 65536;
constexpr size_t XOFF_OFF = TMO_OFF + 4096;
constexpr size_t CNT_OFF = 131072;
constexpr size_t PAD_OFF = 262144 - 4096;
constexpr size_t X_OFF = 262144, X_STRIDE = XBYTES + 4096;
constexpr size_t WS_NEED = X_OFF + 2 * X_STRIDE;
constexpr unsigned long long DEADLINE_TICKS = 50000000ull;
constexpr int WSLOT = 16384, NSLOT = 8, EBUF = NSLOT * WSLOT;

struct Params {
    const char* x;
    const char* w;
    const float* sx;
    const float* sw;
    float* out;
    char* myws;
    char* ws[NR];
    int rank;
    unsigned srcmap;
};

__device__ __forceinline__ void glds16(unsigned voff, const char* sbase, unsigned lds_dst) {
    unsigned keep;
    asm volatile("s_mov_b32 %0, m0\n\ts_mov_b32 m0, %3\n\ts_nop 0\n\tglobal_load_lds_dwordx4 %1, %2\n\ts_mov_b32 m0, %0"
                 : "=&s"(keep) : "v"(voff), "s"(sbase), "s"(lds_dst) : "memory");
}
struct AOff { unsigned o[4]; };
#define A4_STR "global_load_dwordx4 %0, %4, %8\n\tglobal_load_dwordx4 %1, %5, %8\n\t" \
               "global_load_dwordx4 %2, %6, %8\n\tglobal_load_dwordx4 %3, %7, %8"
__device__ __forceinline__ void load_a4(v4i (&d)[4], const AOff& v, const char* sbase) {
    asm volatile(A4_STR : "=&v"(d[0]), "=&v"(d[1]), "=&v"(d[2]), "=&v"(d[3])
                 : "v"(v.o[0]), "v"(v.o[1]), "v"(v.o[2]), "v"(v.o[3]), "s"(sbase) : "memory");
}
#define A4_IF_STR "s_cmp_eq_u32 %9, 0\n\ts_cbranch_scc1 .Lskip%=\n\t" A4_STR "\n.Lskip%=:"
template <bool FIRST>
__device__ __forceinline__ void load_a4_if(v4i (&d)[4], const AOff& v, const char* sbase, int go) {
    if (FIRST) asm volatile(A4_IF_STR : "=&v"(d[0]), "=&v"(d[1]), "=&v"(d[2]), "=&v"(d[3])
                            : "v"(v.o[0]), "v"(v.o[1]), "v"(v.o[2]), "v"(v.o[3]), "s"(sbase), "s"(go) : "memory", "scc");
    else       asm volatile(A4_IF_STR : "+v"(d[0]), "+v"(d[1]), "+v"(d[2]), "+v"(d[3])
                            : "v"(v.o[0]), "v"(v.o[1]), "v"(v.o[2]), "v"(v.o[3]), "s"(sbase), "s"(go) : "memory", "scc");
}
__device__ __forceinline__ int frag_off(int w, int q, unsigned lay) {
    return (((w & 1) << (lay & 7)) | (((w >> 1) & 1) << ((lay >> 4) & 7)) | (((w >> 2) & 1) << ((lay >> 8) & 7)) |
            ((q & 1) << ((lay >> 12) & 7)) | (((q >> 1) & 1) << ((lay >> 16) & 7))) << 10;
}
#define WAIT_A(n, lo, hi) asm volatile("s_waitcnt vmcnt(%2)" : "+v"(lo), "+v"(hi) : "i"(n) : "memory")
#define WAIT_A4(n, a) asm volatile("s_waitcnt vmcnt(%4)" : "+v"(a[0]), "+v"(a[1]), "+v"(a[2]), "+v"(a[3]) : "i"(n) : "memory")

#define STAMP(i) do { } while (0)

__global__ void __launch_bounds__(512, 2) ag_gemm_kernel(Params p) {
    STAMP(0);
    __shared__ __attribute__((aligned(16))) char smem[EBUF + 32768];
    const int tid = threadIdx.x, lane = tid & 63, b = blockIdx.x;
    const int wid = __builtin_amdgcn_readfirstlane(tid >> 6);
    char* myws = p.myws;
    const int rank = p.rank & 7;
    const char* xp = p.x;
    const char* wp = p.w;
    float* outp = p.out;
    const unsigned srcmap = p.srcmap;
    asm volatile("" :: "s"(myws), "s"(rank), "s"(xp), "s"(wp), "s"(outp), "s"(p.sx), "s"(p.sw), "s"(srcmap));
    typedef const float __attribute__((address_space(4))) * cfp;
    const float sxv = *(cfp)(uintptr_t)p.sx, swv = *(cfp)(uintptr_t)p.sw;
    const int idx = b >> 3, mt = idx & 1, nt = idx >> 1;

    unsigned* cntp = (unsigned*)(myws + CNT_OFF) + b * 32;
    unsigned call5 = 0;
    unsigned long long wsl = 0;
    const int shard = (b & 6) ? (b & 7) : (b & 1) ^ URG, cm = (idx >> 2) & 1;
    const unsigned lds0 = (unsigned)(uintptr_t)(LAS char*)smem;
    unsigned wvoff;
    { const int r = tid >> 3, c = tid & 7, f = ((r >> 1) & 3) | (((r >> 5) & 1) << 2); wvoff = (unsigned)(r * NL + ((c ^ f) << 4)); }
    const char* wbase = wp + nt * 128;
    auto stage_w = [&](int ks) {
        const char* g = wbase + (size_t)ks * (128 * NL);
        const unsigned d = lds0 + (unsigned)(ks & (NSLOT - 1)) * WSLOT + (unsigned)wid * 1024u;
        glds16(wvoff, g, d);
        glds16(wvoff, g + 64 * NL, d + 8192u);
    };
    if (__builtin_expect(wid < 4, 1)) {
        const unsigned cnt_raw = __hip_atomic_load(cntp, __ATOMIC_RELAXED, __HIP_MEMORY_SCOPE_AGENT);
        const int v = shard * 4 + (idx >> 3), sub = wid & 1;
        const int rb = v < 24 ? sub : (v - 24) >> 2, ks = v < 24 ? v : 24 + ((v & 3) << 1) + sub;
        const int row = cm * 256 + (idx & 3) * 64 + (wid >> 1) * 32 + rb * 16 + (lane & 15), kk = ks * 128 + 32 * (lane >> 4);
        const v4i* s = (const v4i*)(xp + (size_t)row * KDIM + kk);
        const v4i xv0 = s[0], xv1 = s[1];
        char* dst = myws + X_OFF + (size_t)((cnt_raw + 1u) & 1u) * X_STRIDE + (cm * 32 + v) * 32768 + lane * 16;
        const int cw = (idx & 3) * 2 + (wid >> 1);
        asm volatile("global_store_dwordx4 %0, %2, off sc0 sc1\n\tglobal_store_dwordx4 %1, %3, off sc0 sc1\n\ts_nop 1"
                     :: "v"(dst + frag_off(cw, sub * 2, LAYV)), "v"(dst + frag_off(cw, sub * 2 + 1, LAYV)), "v"(xv0), "v"(xv1) : "memory");
        STAMP(1);
        asm volatile("s_waitcnt vmcnt(0)" ::: "memory");
        if (shard == 0) __builtin_amdgcn_s_barrier();
    }
    asm volatile("" ::: "memory");
    if (__builtin_expect(wid == 4, 0)) {
        const char* gw = p.ws[__builtin_amdgcn_s_getreg((31 << 11) | 20) & 7u];
        unsigned t[10];
#pragma unroll
        for (int i = 0; i < 8; ++i) asm volatile("s_load_dword %0, %1, 0x0" : "=s"(t[i]) : "s"(p.ws[i] + PAD_OFF));
        asm volatile("s_load_dword %0, %1, 0x0" : "=s"(t[8]) : "s"(gw + PAD_OFF + X_STRIDE));
        asm volatile("s_load_dword %0, %1, 0x0" : "=s"(t[9]) : "s"(gw + PAD_OFF + 2 * X_STRIDE));
        if (shard == 0) __builtin_amdgcn_s_barrier();
        __builtin_amdgcn_s_barrier();
        asm volatile("s_waitcnt lgkmcnt(0)" :: "s"(t[0]), "s"(t[1]), "s"(t[2]), "s"(t[3]), "s"(t[4]), "s"(t[5]), "s"(t[6]), "s"(t[7]), "s"(t[8]), "s"(t[9]));
    } else if (__builtin_expect(wid == 7, 0)) {
        const char* go = (const char*)outp + (size_t)((srcmap >> ((__builtin_amdgcn_s_getreg((31 << 11) | 20) & 7u) * 4u)) & 7u) * (MS * NL * 4);
        unsigned t[7];
#pragma unroll
        for (int i = 0; i < 4; ++i) asm volatile("s_load_dword %0, %1, 0x0" : "=s"(t[i]) : "s"(wp + (size_t)i * XBYTES + XBYTES - 64));
#pragma unroll
        for (int i = 0; i < 3; ++i) asm volatile("s_load_dword %0, %1, 0x0" : "=s"(t[4 + i]) : "s"(go + (size_t)i * (XBYTES - 64)));
        if (shard == 0) __builtin_amdgcn_s_barrier();
        __builtin_amdgcn_s_barrier();
        asm volatile("s_waitcnt lgkmcnt(0)" :: "s"(t[0]), "s"(t[1]), "s"(t[2]), "s"(t[3]), "s"(t[4]), "s"(t[5]), "s"(t[6]));
    } else if (__builtin_expect(wid == 6, 0)) {
        wsl = (unsigned long long)(p.ws[lane & 7] + FLAG_OFF + (rank * 16 + cm * 8 + shard) * 128);
        asm volatile("" :: "v"(wsl));
        if (shard == 0) {
            __builtin_amdgcn_s_barrier();
            if (lane < NR) (void)__hip_atomic_fetch_add((GAS unsigned*)wsl, 1u, __ATOMIC_RELAXED, __HIP_MEMORY_SCOPE_SYSTEM);
        }
    }
    STAMP(2);
    LAS unsigned* badp = (LAS unsigned*)(smem + EBUF + 5 * 4096);
    unsigned* tmo = (unsigned*)(myws + TMO_OFF);
    if (wid == 5) {
        call5 = (unsigned)__builtin_amdgcn_readfirstlane(__hip_atomic_load(cntp, __ATOMIC_RELAXED, __HIP_MEMORY_SCOPE_AGENT)) + 1u;
        unsigned xoff = (__builtin_amdgcn_s_getreg((31 << 11) | 20) - (unsigned)b) & 7u;
        if (b == URG)
            __hip_atomic_store((unsigned*)(myws + XOFF_OFF) + lane * 32, call5 * 8u + xoff, __ATOMIC_RELAXED, __HIP_MEMORY_SCOPE_AGENT);
        if (shard == 0) __builtin_amdgcn_s_barrier();
        const unsigned need = call5 * 16u;
        const unsigned long long t0 = __builtin_amdgcn_s_memrealtime();
        unsigned bad = 0, polls = 0, src5;
        unsigned long long sws;
        for (;;) {
            src5 = (srcmap >> (((b + xoff) & 7u) * 4u)) & 7u;
            sws = (unsigned long long)p.ws[(b + xoff) & 7u];
            const unsigned* fl = (const unsigned*)(myws + (lane == 0 ? FLAG_OFF + (src5 * 16 + mt * 8) * 128 : XOFF_OFF + (b >> 2) * 128));
            const unsigned t = __hip_atomic_load(fl, __ATOMIC_RELAXED, __HIP_MEMORY_SCOPE_SYSTEM);
            const unsigned v = (unsigned)__builtin_amdgcn_readlane((int)t, 0), xw = (unsigned)__builtin_amdgcn_readlane((int)t, 1);
            asm volatile("" : "+s"(sws));
            if ((xw >> 3) == (call5 & 0x1fffffffu)) {
                if ((xw & 7u) != xoff) { xoff = xw & 7u; continue; }
                if (v >= need) break;
            }
            if ((++polls & 63u) == 0u) {
                if (__hip_atomic_load(tmo, __ATOMIC_RELAXED, __HIP_MEMORY_SCOPE_AGENT) == call5 ||
                    __builtin_amdgcn_s_memrealtime() - t0 > DEADLINE_TICKS) {
                    __hip_atomic_store(tmo, call5, __ATOMIC_RELAXED, __HIP_MEMORY_SCOPE_AGENT);
                    bad = 8; break;
                }
            }
            __builtin_amdgcn_s_sleep(POLLSLEEP);
        }
        if (lane == 0) { badp[0] = bad | src5; badp[1] = call5; badp[2] = (unsigned)sws; badp[3] = (unsigned)(sws >> 32); }
        asm volatile("s_waitcnt lgkmcnt(0)" ::: "memory");
    }
    const float scale = __builtin_bit_cast(float, __builtin_amdgcn_readfirstlane(__builtin_bit_cast(int, sxv * swv)));
    asm volatile("" :: "s"(scale));
    if (wid != 4 && wid != 7) __builtin_amdgcn_s_barrier();
    asm volatile("" ::: "memory");
    if (wid == 6 && shard != 0 && lane < NR)
        (void)__hip_atomic_fetch_add((GAS unsigned*)wsl, 1u, __ATOMIC_RELAXED, __HIP_MEMORY_SCOPE_SYSTEM);
    if (wid == 5 && lane == 0) __hip_atomic_store(cntp, call5, __ATOMIC_RELAXED, __HIP_MEMORY_SCOPE_AGENT);
    const unsigned state = __builtin_amdgcn_readfirstlane(badp[0]);
    const unsigned call = __builtin_amdgcn_readfirstlane(badp[1]);
    bool bad = state >= 8u;
    const int src = (int)(state & 7u);
    const char* srcws = (const char*)(((unsigned long long)(unsigned)__builtin_amdgcn_readfirstlane(badp[3]) << 32) |
                                      (unsigned)__builtin_amdgcn_readfirstlane(badp[2]));
    STAMP(3);

    v4f acc[2][8];
#pragma unroll
    for (int r = 0; r < 2; ++r)
#pragma unroll
        for (int n = 0; n < 8; ++n) acc[r][n] = (v4f){0.f, 0.f, 0.f, 0.f};

    auto epi_prep = [&](v4f (&c)[8], v4i (&y)[8], int h) {
        const float sc = bad ? __builtin_nanf("") : scale;
        LAS char* ebuf = (LAS char*)smem + EBUF + wid * 4096;
#pragma unroll
        for (int n = 0; n < 4; ++n) {
            const v4f z = __builtin_elementwise_maximum(c[h * 4 + n] * sc, (v4f){0.f, 0.f, 0.f, 0.f});
            *(LAS v4f*)(ebuf + (lane & 15) * 256 + (((n * 4 + (lane >> 4)) ^ (lane & 7)) << 4)) = z;
        }
#pragma unroll
        for (int i = 0; i < 4; ++i) {
            const int row = 4 * i + (lane >> 4);
            y[h * 4 + i] = *(LAS v4i*)(ebuf + row * 256 + (((lane & 15) ^ (row & 7)) << 4));
        }
    };
    auto epi_store = [&](v4i (&y)[8], int r, int j0, int j1) {
        const char* obase = (const char*)(outp + (size_t)(src * MS + mt * 256 + wid * 32 + r * 16) * NL + nt * 128);
        const unsigned ooff = (unsigned)((lane >> 4) * (NL * 4) + (lane & 15) * 16);
#pragma unroll
        for (int j = 0; j < 8; ++j) if (j >= j0 && j < j1)
            asm volatile("global_store_dwordx4 %0, %1, %2 offset:%3 " OSTORE "\n\ts_nop 1"
                         :: "v"(ooff), "v"(y[j]), "s"(obase + (j & 3) * 4 * NL * 4), "i"((j >> 2) * 256) : "memory");
        __builtin_amdgcn_sched_barrier(0);
    };
    auto epilogue = [&](v4f (&c)[8], int r) { v4i y[8]; epi_prep(c, y, 0); epi_store(y, r, 0, 4); epi_prep(c, y, 1); epi_store(y, r, 4, 8); };

    if (!bad) {
        const char* abase = srcws + X_OFF + (size_t)(call & 1u) * X_STRIDE + (size_t)mt * (XBYTES / 2);
        AOff avoff;
#pragma unroll
        for (int q = 0; q < 4; ++q) avoff.o[q] = (unsigned)(frag_off(wid, q, LAYV) + lane * 16);
        v4i ar[4][4];
#define LOAD_AS(slot, k) load_a4(ar[slot], avoff, abase + (size_t)(k) * 32768)
#define LOAD_A(k) LOAD_AS((k) & 3, k)
        LOAD_A(0);
        stage_w(0); stage_w(1);
        __builtin_amdgcn_s_barrier();
        stage_w(2); LOAD_A(1);
        int wn = 3;

        unsigned raddr[8];
        { const int g = lane >> 4, u = lane & 15, q = u >> 1, h = u & 1, f = ((q >> 1) & 3) | ((g & 1) << 2);
#pragma unroll
          for (int n = 0; n < 8; ++n) raddr[n] = lds0 + (unsigned)((32 * g + q) * 128 + ((n ^ f) << 4) + h * 8); }
        auto flip = [&]() {
#pragma unroll
            for (int n = 0; n < 8; ++n) raddr[n] ^= 65536u;
        };
        auto wfrag = [&](int n, unsigned off) {
            v2i t[4];
#pragma unroll
            for (int i = 0; i < 4; ++i)
                t[i] = __builtin_amdgcn_ds_read_tr8_b64_v2i32((LAS v2i*)(uintptr_t)(raddr[n] + off + (unsigned)(i * 1024)));
            return (v8i){t[0][0], t[0][1], t[1][0], t[1][1], t[2][0], t[2][1], t[3][0], t[3][1]};
        };
        auto compute = [&](v4i (&a)[4], int slot4) {
            const v8i x0 = {a[0][0], a[0][1], a[0][2], a[0][3], a[1][0], a[1][1], a[1][2], a[1][3]};
            const v8i x1 = {a[2][0], a[2][1], a[2][2], a[2][3], a[3][0], a[3][1], a[3][2], a[3][3]};
            v8i wf[8];
#pragma unroll
            for (int n = 0; n < 8; ++n) wf[n] = wfrag(n, slot4 * WSLOT);
#pragma unroll
            for (int n = 0; n < 8; ++n) {
                acc[0][n] = __builtin_amdgcn_mfma_scale_f32_16x16x128_f8f6f4(wf[n], x0, acc[0][n], 0, 0, 0, 0, 0, 0);
                acc[1][n] = __builtin_amdgcn_mfma_scale_f32_16x16x128_f8f6f4(wf[n], x1, acc[1][n], 0, 0, 0, 0, 0, 0);
            }
            __builtin_amdgcn_sched_group_barrier(0x100, 12, 0);
#pragma unroll
            for (int n = 0; n < 5; ++n) {
                __builtin_amdgcn_sched_group_barrier(0x8, 2, 0);
                __builtin_amdgcn_sched_group_barrier(0x100, 4, 0);
            }
            __builtin_amdgcn_sched_group_barrier(0x8, 6, 0);
            __builtin_amdgcn_sched_barrier(0);
        };
        auto compute1 = [&](v4f (&c)[8], const v4i& lo, const v4i& hi, int slot4) {
            const v8i x0 = {lo[0], lo[1], lo[2], lo[3], hi[0], hi[1], hi[2], hi[3]};
            v8i wf[8];
#pragma unroll
            for (int n = 0; n < 8; ++n) wf[n] = wfrag(n, slot4 * WSLOT);
#pragma unroll
            for (int n = 0; n < 8; ++n) c[n] = __builtin_amdgcn_mfma_scale_f32_16x16x128_f8f6f4(wf[n], x0, c[n], 0, 0, 0, 0, 0, 0);
            __builtin_amdgcn_sched_group_barrier(0x100, 12, 0);
#pragma unroll
            for (int n = 0; n < 5; ++n) {
                __builtin_amdgcn_sched_group_barrier(0x8, 1, 0);
                __builtin_amdgcn_sched_group_barrier(0x100, 4, 0);
            }
            __builtin_amdgcn_sched_group_barrier(0x8, 3, 0);
            __builtin_amdgcn_sched_barrier(0);
        };
        const char* cbase = myws + FLAG_OFF + (src * 16 + mt * 8) * 128;
        auto ctr_fetch = [&]() {
            unsigned keep;
            asm volatile("s_mov_b32 %0, m0\n\ts_mov_b32 m0, %3\n\ts_nop 0\n\tglobal_load_lds_dword %1, %2 sc0 sc1\n\ts_mov_b32 m0, %0"
                         : "=&s"(keep) : "v"((unsigned)(lane & 7) * 128u), "s"(cbase), "s"(lds0 + EBUF + 4 * 4096) : "memory");
        };
        if (wid == 4) ctr_fetch();
        asm volatile("s_waitcnt vmcnt(8)" ::: "memory");
        const int pre = (PREV >> wid) & 1, post = pre ^ 1;
        for (int k0 = 0; k0 < 24; k0 += 4) {
#pragma unroll
            for (int s = 0; s < 4; ++s) {
                __builtin_amdgcn_s_barrier();
                WAIT_A4(6, ar[s]);
                if (pre) { if (s == 1 && k0 >= 4 && k0 < 20) stage_w(wn++); stage_w(wn++); }
                load_a4_if<true>(ar[(s + 2) & 3], avoff, abase + (size_t)(k0 + s + 2) * 32768, pre);
                __builtin_amdgcn_sched_barrier(0);
                compute(ar[s], s);
                if (s == 1 && k0 < 8 && wid == 4) {
                    const unsigned need = call * 16u;
                    const int sh = lane & (k0 ? 7 : 1);
                    if (!__all(*(LAS unsigned*)(smem + EBUF + 4 * 4096 + sh * 4) >= need)) {
                        const unsigned* fl = (const unsigned*)cbase + sh * 32;
                        const unsigned long long t0 = __builtin_amdgcn_s_memrealtime();
                        while (!__all(__hip_atomic_load(fl, __ATOMIC_RELAXED, __HIP_MEMORY_SCOPE_SYSTEM) >= need)) {
                            if (__hip_atomic_load(tmo, __ATOMIC_RELAXED, __HIP_MEMORY_SCOPE_AGENT) == call ||
                                __builtin_amdgcn_s_memrealtime() - t0 > DEADLINE_TICKS) {
                                __hip_atomic_store(tmo, call, __ATOMIC_RELAXED, __HIP_MEMORY_SCOPE_AGENT);
                                if (lane == 0) badp[0] = 8u;
                                break;
                            }
                            __builtin_amdgcn_s_sleep(2);
                        }
                    }
                    asm volatile("s_waitcnt lgkmcnt(0)" ::: "memory");
                }
                if (post) { if (s == 1 && k0 >= 4 && k0 < 20) stage_w(wn++); stage_w(wn++); }
                load_a4_if<false>(ar[(s + 2) & 3], avoff, abase + (size_t)(k0 + s + 2) * 32768, post);
                if (s == 3 && k0 == 0 && wid == 4) ctr_fetch();
            }
            flip();
        }
        STAMP(10);
        bad = (unsigned)__builtin_amdgcn_readfirstlane(badp[0]) >= 8u;
#define SA 2
#define SB 4
        constexpr int SC = 8 - SA - SB;
#define TAILPRE 0
#define TAIL2(c, a, s0, kn, W31) do { \
            if (TAILPRE == 3) { if (W31 && pre) stage_w(31); load_a4_if<true>(ar[(kn) & 3], avoff, abase + (size_t)(kn) * 32768, pre); __builtin_amdgcn_sched_barrier(0); } \
            if (TAILPRE == 2) { if (W31) stage_w(31); LOAD_A(kn); __builtin_amdgcn_sched_barrier(0); } \
            compute1(c, a[0], a[1], s0); \
            if (TAILPRE == 1) { if (W31) stage_w(31); LOAD_A(kn); __builtin_amdgcn_sched_barrier(0); } \
            if (TAILPRE == 4) { if (W31 && pre) stage_w(31); load_a4_if<true>(ar[(kn) & 3], avoff, abase + (size_t)(kn) * 32768, pre); __builtin_amdgcn_sched_barrier(0); } \
            compute1(c, a[2], a[3], s0 + 1); \
            if (TAILPRE == 0) { if (W31) stage_w(31); LOAD_A(kn); } \
            if (TAILPRE >= 3) { if (W31 && post) stage_w(31); load_a4_if<false>(ar[(kn) & 3], avoff, abase + (size_t)(kn) * 32768, post); } \
        } while (0)
        __builtin_amdgcn_s_barrier(); WAIT_A4(6, ar[0]); TAIL2(acc[0], ar[0], 0, 26, true);
        __builtin_amdgcn_s_barrier(); WAIT_A4(6, ar[1]); TAIL2(acc[0], ar[1], 2, 27, false);
        flip();
        __builtin_amdgcn_s_barrier(); WAIT_A4(4, ar[2]); TAIL2(acc[0], ar[2], 0, 28, false);
        __builtin_amdgcn_s_barrier(); WAIT_A4(4, ar[3]); TAIL2(acc[0], ar[3], 2, 29, false);
        v4i y0[8];
        epi_prep(acc[0], y0, 0); epi_prep(acc[0], y0, 1);
        epi_store(y0, 0, 0, SA);
        flip();
        WAIT_A4(4 + SA, ar[0]); TAIL2(acc[1], ar[0], 0, 30, false);
        epi_store(y0, 0, SA, SA + SB);
        WAIT_A4(4 + SA + SB, ar[1]); TAIL2(acc[1], ar[1], 2, 31, false);
        epi_store(y0, 0, SA + SB, 8);
        flip();
#define TSTAMP(i) do { } while (0)
        TSTAMP(4);
        WAIT_A(6 + SB + SC, ar[2][0], ar[2][1]); TSTAMP(5); compute1(acc[1], ar[2][0], ar[2][1], 0);
        WAIT_A(4 + SB + SC, ar[2][2], ar[2][3]); compute1(acc[1], ar[2][2], ar[2][3], 1);
        TSTAMP(6);
        WAIT_A(2 + SC, ar[3][0], ar[3][1]); TSTAMP(7); compute1(acc[1], ar[3][0], ar[3][1], 2);
        TSTAMP(8);
        WAIT_A(SC, ar[3][2], ar[3][3]); TSTAMP(9);     compute1(acc[1], ar[3][2], ar[3][3], 3);
        STAMP(12);
        epilogue(acc[1], 1);
    } else {
        asm volatile("s_waitcnt vmcnt(0)" ::: "memory");
        __builtin_amdgcn_s_barrier();
        epilogue(acc[0], 0);
        epilogue(acc[1], 1);
    }
}
}

static int near_peers(int rank) {
    int dev = -1, ndev = 0;
    if (hipGetDevice(&dev) != hipSuccess || hipGetDeviceCount(&ndev) != hipSuccess || dev != rank || ndev < NR) return -1;
    int node[NR], port_of[256];
    for (int i = 0; i < 256; ++i) port_of[i] = -1;
    for (int r = 0; r < NR; ++r) {
        char bdf[64] = {0}, path[160];
        if (hipDeviceGetPCIBusId(bdf, (int)sizeof bdf, r) != hipSuccess) return -1;
        for (char* c = bdf; *c; ++c) if (*c >= 'A' && *c <= 'Z') *c = (char)(*c - 'A' + 'a');
        snprintf(path, sizeof path, "/sys/bus/pci/devices/%s/xgmi_port_num", bdf);
        FILE* fp = fopen(path, "r");
        if (!fp) return -1;
        unsigned sn, sp, dn, dp;
        node[r] = -1;
        while (fscanf(fp, " %x:%x -> %x:%x", &sn, &sp, &dn, &dp) == 4) {
            node[r] = (int)(sn & 255u);
            if (r == rank) port_of[dn & 255u] = (int)sp;
        }
        fclose(fp);
        if (node[r] < 0) return -1;
    }
    int mask = 0, cnt = 0;
    for (int r = 0; r < NR; ++r) {
        if (r == rank) continue;
        const int pt = port_of[node[r]];
        if (pt < 0) return -1;
        if (pt <= 3) { mask |= 1 << r; ++cnt; }
    }
    return cnt == 3 ? mask : -1;
}

extern "C" void kernel_launch(int rank, int nranks, ncclComm_t comm,
                              hipStream_t stream,
                              void* const* d_in, const int* in_sizes, int n_in,
                              void* d_out, int out_size,
                              const int* in_dtypes, int out_dtype,
                              void* const* d_out_all,
                              void* const* d_ws_all, size_t ws_bytes) {
    if (nranks != NR || n_in < 4 || ws_bytes < WS_NEED || (unsigned)rank >= (unsigned)NR) {
        fprintf(stderr, "unsupported configuration\n");
        return;
    }
    Params p;
    p.x = (const char*)d_in[0];
    p.w = (const char*)d_in[1];
    p.sx = (const float*)d_in[2];
    p.sw = (const float*)d_in[3];
    p.out = (float*)d_out;
    p.rank = rank;
    p.myws = (char*)d_ws_all[rank];
    {
        int near = near_peers(rank);
        (void)hipGetLastError();
        if (near < 0) near = ((0xB2 >> rank) & 1) ? 0xB2 : 0x4D;
        unsigned map = (unsigned)rank, lo = 1, hi = 4;
        for (int i = 1; i < NR; ++i) {
            const unsigned r = (unsigned)(rank + i) & 7u;
            if ((near >> r) & 1) map |= r << (4 * lo++); else map |= r << (4 * hi++);
        }
        p.srcmap = map;
        for (int i = 0; i < NR; ++i) p.ws[i] = (char*)d_ws_all[(map >> (4 * i)) & 7u];
    }
    hipLaunchKernelGGL(ag_gemm_kernel, dim3(NWG), dim3(512), 0, stream, p);
    HIP_CHECK(hipGetLastError());
}
```

```cpp
#include <hip/hip_runtime.h>
#include <rccl/rccl.h>
#include <cstdio>
#include <cstdint>

#define HIP_CHECK(expr) do { hipError_t _e = (expr); if (_e != hipSuccess) { \
    fprintf(stderr, "HIP error %s at %s:%d\n", hipGetErrorString(_e), __FILE__, __LINE__); \
    return; } } while (0)

typedef int   v8i __attribute__((ext_vector_type(8)));
typedef int   v4i __attribute__((ext_vector_type(4)));
typedef int   v2i __attribute__((ext_vector_type(2)));
typedef float v4f __attribute__((ext_vector_type(4)));
#define LAS __attribute__((address_space(3)))
#define GAS __attribute__((address_space(1)))

namespace {
constexpr int NR = 8;
constexpr int KDIM = 4096;
constexpr int NL = 2048;
constexpr int MS = 512;
constexpr int NWG = 256;
constexpr size_t XBYTES = (size_t)MS * KDIM;
constexpr size_t FLAG_OFF = 0;
constexpr size_t TMO_OFF = 65536;
constexpr size_t XOFF_OFF = TMO_OFF + 4096;
constexpr size_t CNT_OFF = 131072;
constexpr size_t PAD_OFF = 262144 - 4096;
constexpr size_t X_OFF = 262144, X_STRIDE = XBYTES + 32768;
constexpr size_t WS_NEED = X_OFF + 2 * X_STRIDE;
constexpr unsigned long long DEADLINE_TICKS = 50000000ull;
constexpr int WSLOT = 16384, NSLOT = 8, EBUF = NSLOT * WSLOT;

struct Params {
    const char* x;
    const char* w;
    const float* sx;
    const float* sw;
    float* out;
    char* myws;
    char* ws[NR];
    int rank;
    unsigned srcmap;
};

__device__ __forceinline__ unsigned xcc_id() { return __builtin_amdgcn_s_getreg((31 << 11) | 20) & 7u; }

__device__ __forceinline__ void glds16(unsigned voff, const char* sbase, unsigned lds_dst) {
    unsigned keep;
    asm volatile("s_mov_b32 %0, m0\n\ts_mov_b32 m0, %3\n\ts_nop 0\n\tglobal_load_lds_dwordx4 %1, %2\n\ts_mov_b32 m0, %0"
                 : "=&s"(keep) : "v"(voff), "s"(sbase), "s"(lds_dst) : "memory");
}
#define A4_STR "global_load_dwordx4 %0, %4, %5\n\tglobal_load_dwordx4 %1, %4, %5 offset:1024\n\t" \
               "global_load_dwordx4 %2, %4, %5 offset:2048\n\tglobal_load_dwordx4 %3, %4, %5 offset:3072"
__device__ __forceinline__ void load_a4(v4i (&d)[4], unsigned voff, const char* sbase) {
    asm volatile(A4_STR : "=&v"(d[0]), "=&v"(d[1]), "=&v"(d[2]), "=&v"(d[3]) : "v"(voff), "s"(sbase) : "memory");
}
#define A4_IF_STR "s_cmp_eq_u32 %6, 0\n\ts_cbranch_scc1 .Lskip%=\n\t" A4_STR "\n.Lskip%=:"
template <bool FIRST>
__device__ __forceinline__ void load_a4_if(v4i (&d)[4], unsigned voff, const char* sbase, int go) {
    if (FIRST) asm volatile(A4_IF_STR : "=&v"(d[0]), "=&v"(d[1]), "=&v"(d[2]), "=&v"(d[3])
                            : "v"(voff), "s"(sbase), "s"(go) : "memory", "scc");
    else       asm volatile(A4_IF_STR : "+v"(d[0]), "+v"(d[1]), "+v"(d[2]), "+v"(d[3])
                            : "v"(voff), "s"(sbase), "s"(go) : "memory", "scc");
}
#define WAIT_A(n, lo, hi) asm volatile("s_waitcnt vmcnt(%2)" : "+v"(lo), "+v"(hi) : "i"(n) : "memory")
#define WAIT_A4(n, a) asm volatile("s_waitcnt vmcnt(%4)" : "+v"(a[0]), "+v"(a[1]), "+v"(a[2]), "+v"(a[3]) : "i"(n) : "memory")
#define MFMA(a, b, c) __builtin_amdgcn_mfma_scale_f32_16x16x128_f8f6f4(a, b, c, 0, 0, 0, 0, 0, 0)

__global__ void __launch_bounds__(512, 2) ag_gemm_kernel(Params p) {
    __shared__ __attribute__((aligned(16))) char smem[EBUF + 32768];
    const int tid = threadIdx.x, lane = tid & 63, b = blockIdx.x;
    const int wid = __builtin_amdgcn_readfirstlane(tid >> 6);
    char* myws = p.myws;
    const int rank = p.rank & 7;
    const char* xp = p.x;
    const char* wp = p.w;
    float* outp = p.out;
    const unsigned srcmap = p.srcmap;
    asm volatile("" :: "s"(myws), "s"(rank), "s"(xp), "s"(wp), "s"(outp), "s"(p.sx), "s"(p.sw), "s"(srcmap));
    typedef const float __attribute__((address_space(4))) * cfp;
    const float sxv = *(cfp)(uintptr_t)p.sx, swv = *(cfp)(uintptr_t)p.sw;
    const int idx = b >> 3, mt = idx & 1, nt = idx >> 1;

    unsigned* cntp = (unsigned*)(myws + CNT_OFF) + b * 32;
    unsigned call5 = 0;
    unsigned long long wsl = 0;
    const int shard = (b & 6) ? (b & 7) : (b & 1) ^ 1, cm = (idx >> 2) & 1;
    if (__builtin_expect(wid < 4, 1)) {
        const unsigned cnt_raw = __hip_atomic_load(cntp, __ATOMIC_RELAXED, __HIP_MEMORY_SCOPE_AGENT);
        const int v = shard * 4 + (idx >> 3), sub = wid & 1;
        const int rb = v < 24 ? sub : (v - 24) >> 2, ks = v < 24 ? v : 24 + ((v & 3) << 1) + sub;
        const int row = cm * 256 + (idx & 3) * 64 + (wid >> 1) * 32 + rb * 16 + (lane & 15), kk = ks * 128 + 32 * (lane >> 4);
        const v4i* s = (const v4i*)(xp + (size_t)row * KDIM + kk);
        const v4i xv0 = s[0], xv1 = s[1];
        char* dst = myws + X_OFF + (size_t)((cnt_raw + 1u) & 1u) * X_STRIDE + ((cm * 32 + v) * 4 + (idx & 3)) * 8192 + wid * 2048 + lane * 16;
        asm volatile("global_store_dwordx4 %0, %1, off sc0 sc1\n\tglobal_store_dwordx4 %0, %2, off offset:1024 sc0 sc1\n\ts_nop 1"
                     :: "v"(dst), "v"(xv0), "v"(xv1) : "memory");
        asm volatile("s_waitcnt vmcnt(0)" ::: "memory");
        if (shard == 0) __builtin_amdgcn_s_barrier();
    }
    asm volatile("" ::: "memory");
    if (__builtin_expect(wid == 4, 0)) {
        const char* gw = p.ws[xcc_id()];
        unsigned t[10];
#pragma unroll
        for (int i = 0; i < 8; ++i) asm volatile("s_load_dword %0, %1, 0x0" : "=s"(t[i]) : "s"(p.ws[i] + PAD_OFF));
        asm volatile("s_load_dword %0, %1, 0x0" : "=s"(t[8]) : "s"(gw + PAD_OFF + X_STRIDE));
        asm volatile("s_load_dword %0, %1, 0x0" : "=s"(t[9]) : "s"(gw + PAD_OFF + 2 * X_STRIDE));
        if (shard == 0) __builtin_amdgcn_s_barrier();
        __builtin_amdgcn_s_barrier();
        asm volatile("s_waitcnt lgkmcnt(0)" :: "s"(t[0]), "s"(t[1]), "s"(t[2]), "s"(t[3]), "s"(t[4]), "s"(t[5]), "s"(t[6]),
                     "s"(t[7]), "s"(t[8]), "s"(t[9]));
    } else if (__builtin_expect(wid == 7, 0)) {
        const char* go = (const char*)outp + (size_t)((srcmap >> (xcc_id() * 4u)) & 7u) * (MS * NL * 4);
        unsigned t[7];
#pragma unroll
        for (int i = 0; i < 4; ++i) asm volatile("s_load_dword %0, %1, 0x0" : "=s"(t[i]) : "s"(wp + (size_t)i * XBYTES + XBYTES - 64));
#pragma unroll
        for (int i = 0; i < 3; ++i) asm volatile("s_load_dword %0, %1, 0x0" : "=s"(t[4 + i]) : "s"(go + (size_t)i * (XBYTES - 64)));
        if (shard == 0) __builtin_amdgcn_s_barrier();
        __builtin_amdgcn_s_barrier();
        asm volatile("s_waitcnt lgkmcnt(0)" :: "s"(t[0]), "s"(t[1]), "s"(t[2]), "s"(t[3]), "s"(t[4]), "s"(t[5]), "s"(t[6]));
    } else if (__builtin_expect(wid == 6, 0)) {
        wsl = (unsigned long long)(p.ws[lane & 7] + FLAG_OFF + (rank * 16 + cm * 8 + shard) * 128);
        asm volatile("" :: "v"(wsl));
        if (shard == 0) {
            __builtin_amdgcn_s_barrier();
            if (lane < NR) (void)__hip_atomic_fetch_add((GAS unsigned*)wsl, 1u, __ATOMIC_RELAXED, __HIP_MEMORY_SCOPE_SYSTEM);
        }
    }
    LAS unsigned* badp = (LAS unsigned*)(smem + EBUF + 5 * 4096);
    unsigned* tmo = (unsigned*)(myws + TMO_OFF);
    if (wid == 5) {
        call5 = (unsigned)__builtin_amdgcn_readfirstlane(__hip_atomic_load(cntp, __ATOMIC_RELAXED, __HIP_MEMORY_SCOPE_AGENT)) + 1u;
        unsigned xoff = (xcc_id() - (unsigned)b) & 7u;
        if (b == 1)
            __hip_atomic_store((unsigned*)(myws + XOFF_OFF) + lane * 32, call5 * 8u + xoff, __ATOMIC_RELAXED, __HIP_MEMORY_SCOPE_AGENT);
        if (shard == 0) __builtin_amdgcn_s_barrier();
        const unsigned need = call5 * 16u;
        const unsigned long long t0 = __builtin_amdgcn_s_memrealtime();
        unsigned bad = 0, polls = 0, src5;
        unsigned long long sws;
        for (;;) {
            src5 = (srcmap >> (((b + xoff) & 7u) * 4u)) & 7u;
            sws = (unsigned long long)p.ws[(b + xoff) & 7u];
            const unsigned* fl = (const unsigned*)(myws + (lane == 0 ? FLAG_OFF + (src5 * 16 + mt * 8) * 128 : XOFF_OFF + (b >> 2) * 128));
            const unsigned t = __hip_atomic_load(fl, __ATOMIC_RELAXED, __HIP_MEMORY_SCOPE_SYSTEM);
            const unsigned v = (unsigned)__builtin_amdgcn_readlane((int)t, 0), xw = (unsigned)__builtin_amdgcn_readlane((int)t, 1);
            asm volatile("" : "+s"(sws));
            if ((xw >> 3) == (call5 & 0x1fffffffu)) {
                if ((xw & 7u) != xoff) { xoff = xw & 7u; continue; }
                if (v >= need) break;
            }
            if ((++polls & 63u) == 0u) {
                if (__hip_atomic_load(tmo, __ATOMIC_RELAXED, __HIP_MEMORY_SCOPE_AGENT) == call5 ||
                    __builtin_amdgcn_s_memrealtime() - t0 > DEADLINE_TICKS) {
                    __hip_atomic_store(tmo, call5, __ATOMIC_RELAXED, __HIP_MEMORY_SCOPE_AGENT);
                    bad = 8; break;
                }
            }
            __builtin_amdgcn_s_sleep(1);
        }
        if (lane == 0) { badp[0] = bad | src5; badp[1] = call5; badp[2] = (unsigned)sws; badp[3] = (unsigned)(sws >> 32); }
        asm volatile("s_waitcnt lgkmcnt(0)" ::: "memory");
    }
    const float scale = __builtin_bit_cast(float, __builtin_amdgcn_readfirstlane(__builtin_bit_cast(int, sxv * swv)));
    const unsigned lds0 = (unsigned)(uintptr_t)(LAS char*)smem;
    unsigned wvoff;
    { const int r = tid >> 3, c = tid & 7, f = ((r >> 1) & 3) | (((r >> 5) & 1) << 2); wvoff = (unsigned)(r * NL + ((c ^ f) << 4)); }
    const char* wbase = wp + nt * 128;
    auto stage_w = [&](int ks) {
        const char* g = wbase + (size_t)ks * (128 * NL);
        const unsigned d = lds0 + (unsigned)(ks & (NSLOT - 1)) * WSLOT + (unsigned)wid * 1024u;
        glds16(wvoff, g, d);
        glds16(wvoff, g + 64 * NL, d + 8192u);
    };
    asm volatile("" :: "s"(scale));
    if (wid != 4 && wid != 7) __builtin_amdgcn_s_barrier();
    asm volatile("" ::: "memory");
    if (wid == 6 && shard != 0 && lane < NR)
        (void)__hip_atomic_fetch_add((GAS unsigned*)wsl, 1u, __ATOMIC_RELAXED, __HIP_MEMORY_SCOPE_SYSTEM);
    if (wid == 5 && lane == 0) __hip_atomic_store(cntp, call5, __ATOMIC_RELAXED, __HIP_MEMORY_SCOPE_AGENT);
    const unsigned state = __builtin_amdgcn_readfirstlane(badp[0]);
    const unsigned call = __builtin_amdgcn_readfirstlane(badp[1]);
    bool bad = state >= 8u;
    const int src = (int)(state & 7u);
    const char* srcws = (const char*)(((unsigned long long)(unsigned)__builtin_amdgcn_readfirstlane(badp[3]) << 32) |
                                      (unsigned)__builtin_amdgcn_readfirstlane(badp[2]));

    v4f acc[2][8];
#pragma unroll
    for (int r = 0; r < 2; ++r)
#pragma unroll
        for (int n = 0; n < 8; ++n) acc[r][n] = (v4f){0.f, 0.f, 0.f, 0.f};

    auto epi_prep = [&](v4f (&c)[8], v4i (&y)[8], int h) {
        const float sc = bad ? __builtin_nanf("") : scale;
        LAS char* ebuf = (LAS char*)smem + EBUF + wid * 4096;
#pragma unroll
        for (int n = 0; n < 4; ++n) {
            const v4f z = __builtin_elementwise_maximum(c[h * 4 + n] * sc, (v4f){0.f, 0.f, 0.f, 0.f});
            *(LAS v4f*)(ebuf + (lane & 15) * 256 + (((n * 4 + (lane >> 4)) ^ (lane & 7)) << 4)) = z;
        }
#pragma unroll
        for (int i = 0; i < 4; ++i) {
            const int row = 4 * i + (lane >> 4);
            y[h * 4 + i] = *(LAS v4i*)(ebuf + row * 256 + (((lane & 15) ^ (row & 7)) << 4));
        }
    };
    auto epi_store = [&](v4i (&y)[8], int r, int h) {
        const char* obase = (const char*)(outp + (size_t)(src * MS + mt * 256 + wid * 32 + r * 16) * NL + nt * 128);
        const unsigned ooff = (unsigned)((lane >> 4) * (NL * 4) + (lane & 15) * 16);
#pragma unroll
        for (int i = 0; i < 4; ++i)
            asm volatile("global_store_dwordx4 %0, %1, %2 offset:%3 sc0 sc1\n\ts_nop 1"
                         :: "v"(ooff), "v"(y[h * 4 + i]), "s"(obase + i * 4 * NL * 4), "i"(h * 256) : "memory");
        __builtin_amdgcn_sched_barrier(0);
    };
    auto epilogue = [&](v4f (&c)[8], int r) { v4i y[8]; epi_prep(c, y, 0); epi_store(y, r, 0); epi_prep(c, y, 1); epi_store(y, r, 1); };

    if (!bad) {
        const char* abase = srcws + X_OFF + (size_t)(call & 1u) * X_STRIDE + (size_t)mt * (XBYTES / 2) + (size_t)wid * 4096;
        const unsigned avoff = (unsigned)lane * 16u;
        v4i ar[4][4];
        load_a4(ar[0], avoff, abase);
        stage_w(0); stage_w(1);
        __builtin_amdgcn_s_barrier();
        stage_w(2); load_a4(ar[1], avoff, abase + 32768);
        int wn = 3;
        const int pre = wid & 1, post = pre ^ 1;

        unsigned raddr[8];
        { const int g = lane >> 4, u = lane & 15, q = u >> 1, h = u & 1, f = ((q >> 1) & 3) | ((g & 1) << 2);
#pragma unroll
          for (int n = 0; n < 8; ++n) raddr[n] = lds0 + (unsigned)((32 * g + q) * 128 + ((n ^ f) << 4) + h * 8); }
        auto flip = [&]() {
#pragma unroll
            for (int n = 0; n < 8; ++n) raddr[n] ^= 65536u;
        };
        auto wfrag = [&](int n, int slot4) {
            v2i t[4];
#pragma unroll
            for (int i = 0; i < 4; ++i)
                t[i] = __builtin_amdgcn_ds_read_tr8_b64_v2i32((LAS v2i*)(uintptr_t)(raddr[n] + (unsigned)(slot4 * WSLOT + i * 1024)));
            return (v8i){t[0][0], t[0][1], t[1][0], t[1][1], t[2][0], t[2][1], t[3][0], t[3][1]};
        };
        auto join = [](const v4i& lo, const v4i& hi) { return (v8i){lo[0], lo[1], lo[2], lo[3], hi[0], hi[1], hi[2], hi[3]}; };
        auto compute = [&](v4i (&a)[4], int slot4) {
            const v8i x0 = join(a[0], a[1]), x1 = join(a[2], a[3]);
            v8i wf[8];
#pragma unroll
            for (int n = 0; n < 8; ++n) wf[n] = wfrag(n, slot4);
#pragma unroll
            for (int n = 0; n < 8; ++n) { acc[0][n] = MFMA(wf[n], x0, acc[0][n]); acc[1][n] = MFMA(wf[n], x1, acc[1][n]); }
            __builtin_amdgcn_sched_group_barrier(0x100, 12, 0);
#pragma unroll
            for (int n = 0; n < 5; ++n) {
                __builtin_amdgcn_sched_group_barrier(0x8, 2, 0);
                __builtin_amdgcn_sched_group_barrier(0x100, 4, 0);
            }
            __builtin_amdgcn_sched_group_barrier(0x8, 6, 0);
            __builtin_amdgcn_sched_barrier(0);
        };
        auto compute1 = [&](v4f (&c)[8], const v4i& lo, const v4i& hi, int slot4) {
            const v8i x0 = join(lo, hi);
            v8i wf[8];
#pragma unroll
            for (int n = 0; n < 8; ++n) wf[n] = wfrag(n, slot4);
#pragma unroll
            for (int n = 0; n < 8; ++n) c[n] = MFMA(wf[n], x0, c[n]);
            __builtin_amdgcn_sched_group_barrier(0x100, 12, 0);
#pragma unroll
            for (int n = 0; n < 5; ++n) {
                __builtin_amdgcn_sched_group_barrier(0x8, 1, 0);
                __builtin_amdgcn_sched_group_barrier(0x100, 4, 0);
            }
            __builtin_amdgcn_sched_group_barrier(0x8, 3, 0);
            __builtin_amdgcn_sched_barrier(0);
        };
        auto compute2 = [&](v4f (&c)[8], v4i (&a)[4], int slot4) {
            const v8i x0 = join(a[0], a[1]), x1 = join(a[2], a[3]);
            v8i wf[16];
#pragma unroll
            for (int n = 0; n < 16; ++n) wf[n] = wfrag(n & 7, slot4 + (n >> 3));
#pragma unroll
            for (int n = 0; n < 16; ++n) c[n & 7] = MFMA(wf[n], n < 8 ? x0 : x1, c[n & 7]);
            __builtin_amdgcn_sched_group_barrier(0x100, 12, 0);
#pragma unroll
            for (int n = 0; n < 13; ++n) {
                __builtin_amdgcn_sched_group_barrier(0x8, 1, 0);
                __builtin_amdgcn_sched_group_barrier(0x100, 4, 0);
            }
            __builtin_amdgcn_sched_group_barrier(0x8, 3, 0);
            __builtin_amdgcn_sched_barrier(0);
        };
        const char* cbase = myws + FLAG_OFF + (src * 16 + mt * 8) * 128;
        auto ctr_fetch = [&]() {
            unsigned keep;
            asm volatile("s_mov_b32 %0, m0\n\ts_mov_b32 m0, %3\n\ts_nop 0\n\tglobal_load_lds_dword %1, %2 sc0 sc1\n\ts_mov_b32 m0, %0"
                         : "=&s"(keep) : "v"((unsigned)(lane & 7) * 128u), "s"(cbase), "s"(lds0 + EBUF + 4 * 4096) : "memory");
        };
        if (wid == 4) ctr_fetch();
        asm volatile("s_waitcnt vmcnt(8)" ::: "memory");
        for (int k0 = 0; k0 < 24; k0 += 4) {
#pragma unroll
            for (int s = 0; s < 4; ++s) {
                __builtin_amdgcn_s_barrier();
                WAIT_A4(6, ar[s]);
                if (pre) { if (s == 1 && k0 >= 4 && k0 < 20) stage_w(wn++); stage_w(wn++); }
                load_a4_if<true>(ar[(s + 2) & 3], avoff, abase + (size_t)(k0 + s + 2) * 32768, pre);
                __builtin_amdgcn_sched_barrier(0);
                compute(ar[s], s);
                if (s == 1 && k0 < 8 && wid == 4) {
                    const unsigned need = call * 16u;
                    const int sh = lane & (k0 ? 7 : 1);
                    if (!__all(*(LAS unsigned*)(smem + EBUF + 4 * 4096 + sh * 4) >= need)) {
                        const unsigned* fl = (const unsigned*)cbase + sh * 32;
                        const unsigned long long t0 = __builtin_amdgcn_s_memrealtime();
                        while (!__all(__hip_atomic_load(fl, __ATOMIC_RELAXED, __HIP_MEMORY_SCOPE_SYSTEM) >= need)) {
                            if (__hip_atomic_load(tmo, __ATOMIC_RELAXED, __HIP_MEMORY_SCOPE_AGENT) == call ||
                                __builtin_amdgcn_s_memrealtime() - t0 > DEADLINE_TICKS) {
                                __hip_atomic_store(tmo, call, __ATOMIC_RELAXED, __HIP_MEMORY_SCOPE_AGENT);
                                if (lane == 0) badp[0] = 8u;
                                break;
                            }
                            __builtin_amdgcn_s_sleep(2);
                        }
                    }
                    asm volatile("s_waitcnt lgkmcnt(0)" ::: "memory");
                }
                if (post) { if (s == 1 && k0 >= 4 && k0 < 20) stage_w(wn++); stage_w(wn++); }
                load_a4_if<false>(ar[(s + 2) & 3], avoff, abase + (size_t)(k0 + s + 2) * 32768, post);
                if (s == 3 && k0 == 0 && wid == 4) ctr_fetch();
            }
            flip();
        }
        bad = (unsigned)__builtin_amdgcn_readfirstlane(badp[0]) >= 8u;
#define TAIL2(c, a, s0, kn, W31) do { \
            if (W31 && pre) stage_w(31); \
            load_a4_if<true>(ar[(kn) & 3], avoff, abase + (size_t)(kn) * 32768, pre); \
            __builtin_amdgcn_sched_barrier(0); \
            compute2(c, a, s0); \
            if (W31 && post) stage_w(31); \
            load_a4_if<false>(ar[(kn) & 3], avoff, abase + (size_t)(kn) * 32768, post); \
        } while (0)
        __builtin_amdgcn_s_barrier(); WAIT_A4(6, ar[0]); TAIL2(acc[0], ar[0], 0, 26, true);
        __builtin_amdgcn_s_barrier(); WAIT_A4(6, ar[1]); TAIL2(acc[0], ar[1], 2, 27, false);
        flip();
        __builtin_amdgcn_s_barrier(); WAIT_A4(4, ar[2]); TAIL2(acc[0], ar[2], 0, 28, false);
        __builtin_amdgcn_s_barrier(); WAIT_A4(4, ar[3]); TAIL2(acc[0], ar[3], 2, 29, false);
        v4i y0[8];
        epi_prep(acc[0], y0, 0); epi_prep(acc[0], y0, 1);
        epi_store(y0, 0, 0);
        flip();
        WAIT_A4(8, ar[0]); TAIL2(acc[1], ar[0], 0, 30, false);
        epi_store(y0, 0, 1);
        WAIT_A4(12, ar[1]); TAIL2(acc[1], ar[1], 2, 31, false);
        flip();
        WAIT_A(10, ar[2][0], ar[2][1]); compute1(acc[1], ar[2][0], ar[2][1], 0);
        WAIT_A(8, ar[2][2], ar[2][3]);  compute1(acc[1], ar[2][2], ar[2][3], 1);
        WAIT_A(2, ar[3][0], ar[3][1]);  compute1(acc[1], ar[3][0], ar[3][1], 2);
        WAIT_A(0, ar[3][2], ar[3][3]);  compute1(acc[1], ar[3][2], ar[3][3], 3);
        epilogue(acc[1], 1);
    } else {
        __builtin_amdgcn_s_barrier();
        epilogue(acc[0], 0);
        epilogue(acc[1], 1);
    }
}

int near_peers(int rank) {
    int dev = -1, ndev = 0;
    if (hipGetDevice(&dev) != hipSuccess || hipGetDeviceCount(&ndev) != hipSuccess || dev != rank || ndev < NR) return -1;
    int node[NR], port_of[256];
    for (int i = 0; i < 256; ++i) port_of[i] = -1;
    for (int r = 0; r < NR; ++r) {
        char bdf[64] = {0}, path[160];
        if (hipDeviceGetPCIBusId(bdf, (int)sizeof bdf, r) != hipSuccess) return -1;
        for (char* c = bdf; *c; ++c) if (*c >= 'A' && *c <= 'Z') *c = (char)(*c - 'A' + 'a');
        snprintf(path, sizeof path, "/sys/bus/pci/devices/%s/xgmi_port_num", bdf);
        FILE* fp = fopen(path, "r");
        if (!fp) return -1;
        unsigned sn, sp, dn, dp;
        node[r] = -1;
        while (fscanf(fp, " %x:%x -> %x:%x", &sn, &sp, &dn, &dp) == 4) {
            node[r] = (int)(sn & 255u);
            if (r == rank) port_of[dn & 255u] = (int)sp;
        }
        fclose(fp);
        if (node[r] < 0) return -1;
    }
    int mask = 0, cnt = 0;
    for (int r = 0; r < NR; ++r) {
        if (r == rank) continue;
        const int pt = port_of[node[r]];
        if (pt < 0) return -1;
        if (pt <= 3) { mask |= 1 << r; ++cnt; }
    }
    return cnt == 3 ? mask : -1;
}
}

extern "C" void kernel_launch(int rank, int nranks, ncclComm_t comm,
                              hipStream_t stream,
                              void* const* d_in, const int* in_sizes, int n_in,
                              void* d_out, int out_size,
                              const int* in_dtypes, int out_dtype,
                              void* const* d_out_all,
                              void* const* d_ws_all, size_t ws_bytes) {
    if (nranks != NR || n_in < 4 || ws_bytes < WS_NEED || (unsigned)rank >= (unsigned)NR ||
        in_sizes[0] != MS * KDIM || in_sizes[1] != KDIM * NL || out_size != NR * MS * NL) {
        fprintf(stderr, "unsupported configuration\n");
        return;
    }
    Params p;
    p.x = (const char*)d_in[0];
    p.w = (const char*)d_in[1];
    p.sx = (const float*)d_in[2];
    p.sw = (const float*)d_in[3];
    p.out = (float*)d_out;
    p.rank = rank;
    p.myws = (char*)d_ws_all[rank];
    {
        int near = near_peers(rank);
        (void)hipGetLastError();
        if (near < 0) near = ((0xB2 >> rank) & 1) ? 0xB2 : 0x4D;
        unsigned map = (unsigned)rank, lo = 1, hi = 4;
        for (int i = 1; i < NR; ++i) {
            const unsigned r = (unsigned)(rank + i) & 7u;
            if ((near >> r) & 1) map |= r << (4 * lo++); else map |= r << (4 * hi++);
        }
        p.srcmap = map;
        for (int i = 0; i < NR; ++i) p.ws[i] = (char*)d_ws_all[(map >> (4 * i)) & 7u];
    }
    hipLaunchKernelGGL(ag_gemm_kernel, dim3(NWG), dim3(512), 0, stream, p);
    HIP_CHECK(hipGetLastError());
}
```
